```python
import numpy as np

kernel_name = "gpu_gemm_ar_m4096_k4096_n4096_f32_relu"


ABSMAX_RTOL = 0.001

COMPUTE_DTYPE = "float32"
EPILOGUE = "relu"
IN_QUANT = None
OUT_QUANT = None
IN_DTYPES = {"x": "float32", "w_mat": "float32"}
WORLD_SIZE = 8

def setup_inputs():

    x = np.random.randn(4096, 4096).astype(np.float32)
    w_mat = (np.random.randn(4096, 4096) / np.sqrt(4096)).astype(np.float32)
    return {"x": x, "w_mat": w_mat}

def _epilogue(y):
    return np.maximum(y, 0.0).astype(np.float32)

def reference(x, w_mat):
    y = (x @ w_mat).astype(np.float32)
    return _epilogue(y)
```

```cpp
#include <hip/hip_runtime.h>
#include <rccl/rccl.h>
#include <cstdio>
#include <cstdint>

typedef unsigned u32x4 __attribute__((ext_vector_type(4)));
typedef float f32x4 __attribute__((ext_vector_type(4)));
typedef float f32x2 __attribute__((ext_vector_type(2)));
typedef _Float16 f16x8 __attribute__((ext_vector_type(8)));
typedef unsigned long long u64;

#define HIP_CHECK(expr) do { hipError_t _e = (expr); if (_e != hipSuccess) { \
    fprintf(stderr, "HIP error %s at %s:%d\n", hipGetErrorString(_e), __FILE__, __LINE__); \
    return; } } while (0)

#define OUT_POL 0
constexpr int NR = 8;
constexpr int GN = 4096;
constexpr int KL = 512;
constexpr int NWG = 256, NTHR = 512;
#define NSEND_ 64
#define GATE_ NSEND_
constexpr int NSEND = NSEND_;
constexpr int NGEMM = 128;
constexpr int NMSG = NGEMM * 8;
constexpr size_t UNIT = 16384;
constexpr size_t SLOT = 9216;
constexpr unsigned long long DEADLINE = 40000000ull;

constexpr size_t OFF_CNT   = 0;
constexpr size_t OFF_TMO   = 4096;
constexpr size_t OFF_TICK  = 6144;
constexpr size_t OFF_SDONE = 8192;
constexpr size_t OFF_XF    = 16384;
constexpr size_t OFF_WF    = 32768;
constexpr size_t OFF_OF    = 65536;
constexpr size_t OFF_DBG   = 262144;
constexpr size_t OFF_X     = 1u << 20;
constexpr size_t OFF_W     = OFF_X + 16 * 64 * UNIT;
constexpr size_t OFF_O     = OFF_W + 8 * 64 * UNIT;
constexpr size_t WS_NEED   = OFF_O + (size_t)NR * NMSG * SLOT;

constexpr int LDS_STAGE = 32768, LDS_MISC = 4 * LDS_STAGE, LDS_BYTES = LDS_MISC + 1024;
constexpr int WAVE_LDS = 16384;

struct Params { const float* x; const float* w; float* out; char* ws[NR]; int rank; };

__device__ __forceinline__ void st_wt(void* p, u32x4 v) {
    asm volatile("global_store_dwordx4 %0, %1, off sc0 sc1\n\ts_nop 1" :: "v"(p), "v"(v) : "memory");
}
__device__ __forceinline__ void glds1(const void* g, unsigned lds) {
    unsigned keep;
    asm volatile("s_mov_b32 %0, m0\n\ts_mov_b32 m0, %2\n\ts_nop 0\n\t"
                 "global_load_lds_dwordx4 %1, off sc0 sc1\n\ts_mov_b32 m0, %0"
                 : "=&s"(keep) : "v"(g), "s"(lds) : "memory");
}
#define GSC " sc0 sc1"
__device__ __forceinline__ void glds4(const void* g, unsigned lds) {
    unsigned keep;
    asm volatile("s_mov_b32 %0, m0\n\ts_mov_b32 m0, %2\n\ts_nop 0\n\t"
                 "global_load_lds_dwordx4 %1, off" GSC "\n\t"
                 "global_load_lds_dwordx4 %1, off offset:1024" GSC "\n\t"
                 "global_load_lds_dwordx4 %1, off offset:2048" GSC "\n\t"
                 "global_load_lds_dwordx4 %1, off offset:3072" GSC "\n\t"
                 "s_mov_b32 m0, %0"
                 : "=&s"(keep) : "v"(g), "s"(lds) : "memory");
}
__device__ __forceinline__ void st_out(void* p, u32x4 v) {
    *(u32x4*)p = v;
}
#define WAIT_V(n) asm volatile("s_waitcnt vmcnt(" #n ")" ::: "memory")
#define WAIT_L(n) asm volatile("s_waitcnt lgkmcnt(" #n ")" ::: "memory")
#define BAR() do { asm volatile("s_waitcnt lgkmcnt(0)" ::: "memory"); __builtin_amdgcn_s_barrier(); asm volatile("" ::: "memory"); } while (0)

__device__ __forceinline__ u64 ld_sys64(const void* p) {
    return __hip_atomic_load((const u64*)p, __ATOMIC_RELAXED, __HIP_MEMORY_SCOPE_SYSTEM);
}
__device__ __forceinline__ void st_sys64(void* p, u64 v) {
    __hip_atomic_store((u64*)p, v, __ATOMIC_RELAXED, __HIP_MEMORY_SCOPE_SYSTEM);
}
__device__ __forceinline__ unsigned ld_ag32(const void* p) {
    return __hip_atomic_load((const unsigned*)p, __ATOMIC_RELAXED, __HIP_MEMORY_SCOPE_AGENT);
}
__device__ __forceinline__ void st_ag32(void* p, unsigned v) {
    __hip_atomic_store((unsigned*)p, v, __ATOMIC_RELAXED, __HIP_MEMORY_SCOPE_AGENT);
}
__device__ __forceinline__ bool expired(char* me, unsigned call, u64 t0) {
    bool e = ld_ag32(me + OFF_TMO) == call || __builtin_amdgcn_s_memrealtime() - t0 > DEADLINE;
    e = __builtin_amdgcn_readfirstlane(e);
    if (e) st_ag32(me + OFF_TMO, call);
    return e;
}
__device__ __forceinline__ void wave_scan6(unsigned c, unsigned& pre, unsigned& tot) {
    pre = 0; tot = 0;
#pragma unroll
    for (int b = 0; b < 6; ++b) {
        u64 m = __builtin_amdgcn_ballot_w64((c >> b) & 1);
        unsigned p = __builtin_amdgcn_mbcnt_hi((unsigned)(m >> 32), __builtin_amdgcn_mbcnt_lo((unsigned)m, 0));
        pre += p << b;
        tot += (unsigned)__builtin_popcountll(m) << b;
    }
}
__device__ __forceinline__ void tile_of(int g, int& i, int& j) {
    int xcd = g & 7, t = g >> 3;
    i = 2 * (t >> 1) + (xcd >> 2);
    j = 2 * (xcd & 3) + (t & 1);
}
__device__ __forceinline__ int unpacker_of(int msg, int src) { return (msg + 136 * src) & (NMSG - 1); }

#define STAMP(k) do {} while (0)

#define LAG 4
__device__ __forceinline__ void unit_of(int U, bool& isx, int& grp, int& strip, int& ks) {
    bool late; int o;
    if (U < 40 * LAG) { ks = U / 40; o = U - 40 * ks; late = false; }
    else if (U < 40 * LAG + 64 * (8 - LAG)) {
        const int V = U - 40 * LAG, s = V >> 6; o = V & 63;
        late = o < 24;
        if (late) ks = s; else { ks = s + LAG; o -= 24; }
    } else {
        const int V = U - (40 * LAG + 64 * (8 - LAG)), s = V / 24;
        o = V - 24 * s; ks = 8 - LAG + s; late = true;
    }
    isx = late || o < 8;
    if (isx) { grp = o & 1; strip = (late ? 4 : 0) + (o >> 1); }
    else     { grp = ((o >> 3) + o) & 3; strip = o & 7; }
}
__device__ __forceinline__ void unit_load(const Params& P, int U, int wv, int lane, f32x2 (&f)[8]) {
    bool isx; int grp, strip, ks; unit_of(U, isx, grp, strip, ks);
    if (isx) {
        int kg = (lane & 7) ^ (lane >> 3);
        const float* s = P.x + (size_t)(2048 * grp + 128 * strip + 16 * wv + (lane >> 3)) * KL + 64 * ks + 4 * kg;
#pragma unroll
        for (int t = 0; t < 2; ++t) {
            f32x4 a = *(const f32x4*)(s + t * 8 * KL), b = *(const f32x4*)(s + t * 8 * KL + 32);
            f[4 * t] = a.xy; f[4 * t + 1] = a.zw; f[4 * t + 2] = b.xy; f[4 * t + 3] = b.zw;
        }
    } else {
        const float* s = P.w + (size_t)(64 * ks + 4 * wv) * GN + 1024 * grp + 128 * strip + 4 * (lane & 31) + 2 * (lane >> 5);
#pragma unroll
        for (int e = 0; e < 8; ++e) {
            f[e] = *(const f32x2*)(s + (size_t)((e & 3) + (e >> 2) * 32) * GN);
        }
    }
}

__global__ __launch_bounds__(NTHR, 2) void gemm_ar_fused(Params P) {
    extern __shared__ __attribute__((aligned(16))) char smem[];
    const int tid = threadIdx.x, lane = tid & 63;
    const int wv = __builtin_amdgcn_readfirstlane(tid >> 6);
    const int b = blockIdx.x, rank = P.rank;
    char* const me = P.ws[rank];
    const unsigned lds0 = (unsigned)(uintptr_t)(__attribute__((address_space(3))) char*)smem;
    const u64 t0 = __builtin_amdgcn_s_memrealtime();
    STAMP(0);

    f32x2 f[8];
    if (b < NSEND) unit_load(P, b, wv, lane, f);

    unsigned* misc = (unsigned*)(smem + LDS_MISC);
    if (tid == 0) {
        unsigned c = ld_ag32(me + OFF_CNT + 4 * b) + 1;
        st_ag32(me + OFF_CNT + 4 * b, c);
        misc[0] = c; misc[1] = 0;
    }
    __syncthreads();
    const unsigned call = __builtin_amdgcn_readfirstlane(misc[0]);

    if (b >= NSEND && b < NSEND + NGEMM) {
        const int g = b - NSEND;
        int ti, tj; tile_of(g, ti, tj);
        const size_t fo = (size_t)((rank + lane) & 7) * 128 + (lane >> 3) * 8;
        const char* fw = me + OFF_WF + (size_t)tj * 1024 + fo;
        const char* fx = me + OFF_XF + (size_t)ti * 1024 + fo;
        u64 ready = 0;
        auto need = [&](int kk) {
            while (!((ready >> kk) & 1)) {
                const u64 a = ld_sys64(fw), c = ld_sys64(fx);
                ready = __builtin_amdgcn_ballot_w64((unsigned)(a >> 32) >= call && (unsigned)(c >> 32) >= call);
                if ((ready >> kk) & 1) break;
                if (expired(me, call, t0)) { ready = ~0ull; break; }
                __builtin_amdgcn_s_sleep(12);
            }
        };
        if (wv == 0) { need(0); need(1); need(2); }
        BAR();
        STAMP(1);
        const int wr = wv >> 1, wc = wv & 1, q = lane >> 4, r = lane & 15;
        f32x4 acc[2][4];
#pragma unroll
        for (int mi = 0; mi < 2; ++mi)
#pragma unroll
            for (int nn = 0; nn < 4; ++nn) acc[mi][nn] = (f32x4){0.f, 0.f, 0.f, 0.f};
        {
            const char* gsrc = (wv < 4 ? me + OFF_X + (size_t)ti * 64 * UNIT : me + OFF_W + (size_t)tj * 64 * UNIT)
                               + (wv & 3) * 4096 + lane * 16;
            const unsigned ldst = lds0 + wv * 4096;
            auto stage = [&](int kk) {
                glds4(gsrc + (size_t)(((rank + kk) & 7) * 8 + (kk >> 3)) * UNIT, ldst + (kk & 3) * LDS_STAGE);
            };
            stage(0); stage(1); stage(2);
            const int aoff = (32 * wr + r) * 128, boff = 16384 + q * 2048 + (16 * wc + r) * 16;
            for (int kk = 0; kk < 64; ++kk) {
                if (wv == 0 && kk + 3 < 64) need(kk + 3);
                if (kk < 62) WAIT_V(8); else if (kk == 62) WAIT_V(4); else WAIT_V(0);
                BAR();
                if (kk + 3 < 64) stage(kk + 3);
                const char* st = smem + (kk & 3) * LDS_STAGE;
#pragma unroll
                for (int s = 0; s < 2; ++s) {
                    f16x8 af[2], bf[4];
#pragma unroll
                    for (int mi = 0; mi < 2; ++mi)
                        af[mi] = *(const f16x8*)(st + aoff + mi * 2048 + (((4 * s + q) ^ (r & 7)) * 16));
#pragma unroll
                    for (int nn = 0; nn < 4; ++nn)
                        bf[nn] = *(const f16x8*)(st + boff + s * 8192 + (nn & 1) * 1024 + (nn >> 1) * 512);
#pragma unroll
                    for (int mi = 0; mi < 2; ++mi)
#pragma unroll
                        for (int nn = 0; nn < 4; ++nn)
                            acc[mi][nn] = __builtin_amdgcn_mfma_f32_16x16x32_f16(af[mi], bf[nn], acc[mi][nn], 0, 0, 0);
                }
            }
        }
        BAR();
        if (ld_ag32(me + OFF_TMO) == call) {
#pragma unroll
            for (int mi = 0; mi < 2; ++mi)
#pragma unroll
                for (int nn = 0; nn < 4; ++nn) acc[mi][nn] = (f32x4){NAN, NAN, NAN, NAN};
        }
        STAMP(2);
        unsigned u[32], mask = 0, umax = 0, umin1 = 0xffffffffu;
#pragma unroll
        for (int mi = 0; mi < 2; ++mi)
#pragma unroll
            for (int j = 0; j < 4; ++j)
#pragma unroll
                for (int nn = 0; nn < 4; ++nn) {
                    const int e = (mi * 4 + j) * 4 + nn;
                    const float v = acc[mi][nn][j];
                    const bool keep = !(v <= 0.f);
                    u[e] = keep ? __float_as_uint(v) : 0u;
                    mask |= (keep ? 1u : 0u) << e;
                    umax = max(umax, u[e]);
                    umin1 = min(umin1, u[e] - 1u);
                }
        {
            float* o = P.out + (size_t)(2048 * (rank >> 2) + 128 * ti + 32 * wr + 4 * q) * GN
                       + 1024 * (rank & 3) + 128 * tj + 64 * wc + 4 * r;
#pragma unroll
            for (int e4 = 0; e4 < 8; ++e4)
                st_out(o + (size_t)((e4 >> 2) * 16 + (e4 & 3)) * GN, (u32x4){u[4 * e4], u[4 * e4 + 1], u[4 * e4 + 2], u[4 * e4 + 3]});
        }
#pragma unroll
        for (int o = 32; o > 0; o >>= 1) {
            umax = max(umax, (unsigned)__shfl_xor((int)umax, o));
            umin1 = min(umin1, (unsigned)__shfl_xor((int)umin1, o));
        }
        unsigned pre, n; wave_scan6(__builtin_popcount(mask), pre, n);
        n = __builtin_amdgcn_readfirstlane(n);
        const unsigned ebase = __builtin_amdgcn_readfirstlane((umin1 + 1u) >> 23);
        const bool raw = __builtin_amdgcn_readfirstlane((umax >> 23) - ebase) >= 32u;
        const unsigned sub = raw ? 0u : ebase << 23;
        char* ml = smem + wv * WAVE_LDS;
        *(unsigned*)(ml + lane * 4) = mask;
        {
            unsigned pos = 256 + pre * 4;
#pragma unroll
            for (int e = 0; e < 32; ++e) {
                const bool k = (mask >> e) & 1;
                *(unsigned*)(ml + (k ? pos : 12288u + lane * 4)) = u[e] - sub;
                pos += k ? 4 : 0;
            }
        }
        const unsigned nq = (n + 31) >> 5;
        if (!raw && lane < nq) {
            const u32x4 h = *(const u32x4*)(ml + 256 + (7 * nq + lane) * 16);
#pragma unroll
            for (int p = 0; p < 7; ++p) {
                u32x4* a = (u32x4*)(ml + 256 + (p * nq + lane) * 16);
                *a = (*a & 0x0fffffffu) | ((h >> (4 * p)) << 28);
            }
        }
        const unsigned Q = 16 + (raw ? (n + 3) >> 2 : 7 * nq);
        const u64 fl = ((u64)call << 32) | n | (ebase << 16) | (raw ? 0x80000000u : 0u);
        while (__builtin_popcountll(__builtin_amdgcn_ballot_w64(lane < NSEND && ld_ag32(me + OFF_SDONE + 4 * lane) >= call)) < GATE_) {
            if (expired(me, call, t0)) break;
            __builtin_amdgcn_s_sleep(8);
        }
        STAMP(3);
        const int msg = g * 8 + wv;
        const int uw = unpacker_of(msg, rank);
        for (int t = 0; t < NR - 1; ++t) {
            const unsigned d = (unsigned)(rank + 1 + (msg + t) % (NR - 1)) & (NR - 1);
            char* peer = P.ws[d];
            char* dst = peer + OFF_O + ((size_t)rank * NMSG + msg) * SLOT + lane * 16;
            for (unsigned i = 0; i < Q; i += 64) {
                if (i + lane < Q) st_wt(dst + i * 16, *(const u32x4*)(ml + (i + lane) * 16));
            }
            WAIT_V(0);
            if (lane == 0) st_sys64(peer + OFF_OF + (size_t)uw * 128 + rank * 8, fl);
            STAMP(4 + t);
        }
        return;
    }

    if (b < NSEND) {
        int U = b, it = 0;
        unsigned* tick = (unsigned*)(me + OFF_TICK);
        if (tid == 0) misc[2] = __hip_atomic_fetch_add(tick, 1u, __ATOMIC_RELAXED, __HIP_MEMORY_SCOPE_AGENT);
        __syncthreads();
        while (true) {
            const int Un = NSEND + (int)(__builtin_amdgcn_readfirstlane(misc[2 + (it & 1)]) - (call - 1) * 512u);
            const bool more = (unsigned)Un < 512u;
            f16x8 c0, c1;
            bool isx; int grp, strip, ks; unit_of(U, isx, grp, strip, ks);
            if (isx) {
#pragma unroll
                for (int e = 0; e < 8; ++e) { c0[e] = (_Float16)f[e >> 1][e & 1]; c1[e] = (_Float16)f[4 + (e >> 1)][e & 1]; }
            } else {
#pragma unroll
                for (int e = 0; e < 8; ++e) { c0[e] = (_Float16)f[e][0]; c1[e] = (_Float16)f[e][1]; }
            }
            asm volatile("" : "+v"(c0), "+v"(c1));
            if (more) unit_load(P, Un, wv, lane, f);
            const size_t uoff = (isx ? OFF_X : OFF_W) + (size_t)(strip * 64 + rank * 8 + ks) * UNIT + wv * 2048 + lane * 16;
            const int nd = isx ? 4 : 2;
            for (int t = 0; t < nd; ++t) {
                const unsigned d = (unsigned)(isx ? 4 * grp + ((t + wv) & 3) : grp + 4 * ((t + wv) & 1)) & (NR - 1);
                char* dst = P.ws[d] + uoff;
                st_wt(dst, __builtin_bit_cast(u32x4, c0));
                st_wt(dst + 1024, __builtin_bit_cast(u32x4, c1));
            }
            if (tid == 0) {
                if (more) misc[2 + ((it + 1) & 1)] = __hip_atomic_fetch_add(tick, 1u, __ATOMIC_RELAXED, __HIP_MEMORY_SCOPE_AGENT);
                else st_ag32(me + OFF_SDONE + 4 * b, call);
            }
            WAIT_V(0);
            BAR();
            if (wv == 7 && lane < nd) {
                const unsigned d = (unsigned)(isx ? 4 * grp + lane : grp + 4 * lane) & (NR - 1);
                st_sys64(P.ws[d] + (isx ? OFF_XF : OFF_WF) + (size_t)(strip * 8 + rank) * 128 + ks * 8, (u64)call << 32);
            }
            if (it == 1) STAMP(2); if (it == 4) STAMP(5);
            if (!more) break;
            U = Un; ++it;
        }
        STAMP(8);
    }

    {
        const int uw = (b < NSEND ? b : b - NGEMM) * 8 + wv;
        char* ml = smem + wv * WAVE_LDS;
        const unsigned mlds = lds0 + wv * WAVE_LDS;
        unsigned todo = 0xffu & ~(1u << rank);
        bool bad = false;
        int cnt = 0;
        while (todo) {
            u64 fl = 0;
            if (lane < NR) fl = ld_sys64(me + OFF_OF + (size_t)uw * 128 + lane * 8);
            unsigned ready = (unsigned)__builtin_amdgcn_ballot_w64((unsigned)(fl >> 32) >= call) & todo;
            if (!ready) {
                if (expired(me, call, t0)) { bad = true; ready = todo; }
                else { __builtin_amdgcn_s_sleep(16); continue; }
            }
            if (ld_ag32(me + OFF_TMO) == call) bad = true;
            while (ready) {
                const int src = __builtin_ctz(ready);
                ready &= ready - 1; todo &= ~(1u << src);
                const unsigned info = __builtin_amdgcn_readlane((unsigned)fl, src);
                const int msg = (uw - 136 * src) & (NMSG - 1);
                int ti, tj; tile_of(msg >> 3, ti, tj);
                const int wr = (msg & 7) >> 1, wc = msg & 1;
                float* o = P.out + (size_t)(2048 * (src >> 2) + 128 * ti + 32 * wr + 4 * (lane >> 4)) * GN
                           + 1024 * (src & 3) + 128 * tj + 64 * wc + 4 * (lane & 15);
                if (bad) {
                    for (int e4 = 0; e4 < 8; ++e4)
                        *(f32x4*)(o + (size_t)((e4 >> 2) * 16 + (e4 & 3)) * GN) = (f32x4){NAN, NAN, NAN, NAN};
                    continue;
                }
                const unsigned n = min(info & 0xfffu, 2048u), nq = (n + 31) >> 5;
                const bool raw = info >> 31;
                const unsigned Q = 16 + (raw ? (n + 3) >> 2 : 7 * nq);
                const char* srcp = me + OFF_O + ((size_t)src * NMSG + msg) * SLOT + lane * 16;
                WAIT_L(0);
                for (unsigned i = 0; i < Q; i += 64) glds1(srcp + i * 16, mlds + i * 16);
                WAIT_V(0);
                if (!raw && lane < nq) {
                    u32x4 h = {0u, 0u, 0u, 0u};
#pragma unroll
                    for (int p = 0; p < 7; ++p)
                        h |= (*(const u32x4*)(ml + 256 + (p * nq + lane) * 16) >> 28) << (4 * p);
                    *(u32x4*)(ml + 256 + (7 * nq + lane) * 16) = h;
                }
                const unsigned mask = *(const unsigned*)(ml + lane * 4);
                unsigned pre, tot; wave_scan6(__builtin_popcount(mask), pre, tot);
                const unsigned am = raw ? 0xffffffffu : 0x0fffffffu, add = raw ? 0u : ((info >> 16) & 0x1ffu) << 23;
                unsigned pos = 256 + pre * 4;
#pragma unroll
                for (int e4 = 0; e4 < 8; ++e4) {
                    u32x4 v;
#pragma unroll
                    for (int c = 0; c < 4; ++c) {
                        const unsigned k = (unsigned)(((int)(mask << (31 - (4 * e4 + c)))) >> 31);
                        v[c] = ((*(const unsigned*)(ml + pos) & am) + add) & k;
                        pos += k & 4u;
                    }
                    st_out(o + (size_t)((e4 >> 2) * 16 + (e4 & 3)) * GN, v);
                }
                STAMP(9 + cnt); ++cnt;
            }
        }
    }
}

extern "C" void kernel_launch(int rank, int nranks, ncclComm_t comm,
                              hipStream_t stream,
                              void* const* d_in, const int* in_sizes, int n_in,
                              void* d_out, int out_size,
                              const int* in_dtypes, int out_dtype,
                              void* const* d_out_all,
                              void* const* d_ws_all, size_t ws_bytes) {
    if (nranks != NR || (unsigned)rank >= (unsigned)NR || ws_bytes < WS_NEED || n_in < 2 ||
        in_sizes[0] != 4096 * KL || in_sizes[1] != KL * GN || out_size != 4096 * GN) {
        fprintf(stderr, "unexpected problem shape\n");
        return;
    }
    HIP_CHECK(hipFuncSetAttribute((const void*)gemm_ar_fused, hipFuncAttributeMaxDynamicSharedMemorySize, LDS_BYTES));
    Params P;
    P.x = (const float*)d_in[0];
    P.w = (const float*)d_in[1];
    P.out = (float*)d_out;
    for (int p = 0; p < NR; ++p) P.ws[p] = (char*)d_ws_all[p];
    P.rank = rank;
    gemm_ar_fused<<<NWG, NTHR, LDS_BYTES, stream>>>(P);
    HIP_CHECK(hipGetLastError());
}
```

```cpp
#include <hip/hip_runtime.h>
#include <rccl/rccl.h>
#include <cstdio>
#include <cstdint>

typedef unsigned u32x4 __attribute__((ext_vector_type(4)));
typedef float f32x4 __attribute__((ext_vector_type(4)));
typedef float f32x2 __attribute__((ext_vector_type(2)));
typedef _Float16 f16x8 __attribute__((ext_vector_type(8)));
typedef unsigned long long u64;

#define HIP_CHECK(expr) do { hipError_t _e = (expr); if (_e != hipSuccess) { \
    fprintf(stderr, "HIP error %s at %s:%d\n", hipGetErrorString(_e), __FILE__, __LINE__); \
    return; } } while (0)

#define OUT_POL 0
constexpr int NR = 8;
constexpr int GN = 4096;
constexpr int KL = 512;
constexpr int NWG = 256, NTHR = 512;
#define NP_ 32
#define NPI_ 28
#define DLY0 40
#define DLYA 150
#define DLYS 4
#define DLYM 600
#define WPB 4
constexpr int NP = NP_;
constexpr int NPI = NPI_;
constexpr int NSEND = (7 * NP + WPB - 1) / WPB;
constexpr int NGEMM = 128;
constexpr int NMSG = NGEMM * 8;
constexpr int NPIECE = 768;
constexpr size_t UNIT = 16384;
constexpr size_t SLOT = 9216;
constexpr unsigned long long DEADLINE = 40000000ull;
#define PUSHB 1
constexpr int NPUSH = 20 * PUSHB;
constexpr int P0 = 48 * PUSHB;
static_assert(NSEND + NPUSH <= 128, "");

constexpr size_t OFF_CNT   = 57344;
constexpr size_t OFF_TMO   = 4096;
constexpr size_t OFF_TICK  = 6144;
constexpr size_t OFF_LF    = 16384;
constexpr size_t OFF_AF    = 49152;
constexpr size_t OFF_JT    = 8192;
constexpr size_t OFF_DBG   = 262144;
constexpr size_t OFF_Q     = 65536;
constexpr size_t OFF_X     = 1u << 20;
constexpr size_t OFF_S     = OFF_X + 24 * 64 * UNIT;
constexpr size_t OFF_O     = OFF_S + 512 * UNIT;
constexpr size_t WS_NEED   = OFF_O + (size_t)NMSG * SLOT;

constexpr int LDS_STAGE = 32768, LDS_MISC = 4 * LDS_STAGE, LDS_BYTES = LDS_MISC + 1024;
constexpr int WAVE_LDS = 16384;

struct Params { const float* x; const float* w; float* out; char* ws[NR]; int rank; };

__device__ __forceinline__ char* ws_sel(const Params& P, int d) {
    char* r = P.ws[0];
#pragma unroll
    for (int p = 1; p < NR; ++p) { char* w = P.ws[p]; asm("" : "+s"(w)); r = d == p ? w : r; }
    return r;
}
__device__ __forceinline__ void st_wt(void* p, u32x4 v) {
    asm volatile("global_store_dwordx4 %0, %1, off sc0 sc1\n\ts_nop 1" :: "v"(p), "v"(v) : "memory");
}
__device__ __forceinline__ void glds1(const void* g, unsigned lds) {
    unsigned keep;
    asm volatile("s_mov_b32 %0, m0\n\ts_mov_b32 m0, %2\n\ts_nop 0\n\t"
                 "global_load_lds_dwordx4 %1, off sc0 sc1\n\ts_mov_b32 m0, %0"
                 : "=&s"(keep) : "v"(g), "s"(lds) : "memory");
}
#define OSC ""
#define STAG 0
#define GSC " sc0 sc1"
__device__ __forceinline__ void glds4(const void* g, unsigned lds) {
    unsigned keep;
    asm volatile("s_mov_b32 %0, m0\n\ts_mov_b32 m0, %2\n\ts_nop 0\n\t"
                 "global_load_lds_dwordx4 %1, off" GSC "\n\t"
                 "global_load_lds_dwordx4 %1, off offset:1024" GSC "\n\t"
                 "global_load_lds_dwordx4 %1, off offset:2048" GSC "\n\t"
                 "global_load_lds_dwordx4 %1, off offset:3072" GSC "\n\t"
                 "s_mov_b32 m0, %0"
                 : "=&s"(keep) : "v"(g), "s"(lds) : "memory");
}
__device__ __forceinline__ void glds4s(const void* g, unsigned lds) {
    unsigned keep;
    asm volatile("s_mov_b32 %0, m0\n\ts_mov_b32 m0, %2\n\ts_nop 0\n\t"
                 "global_load_lds_dwordx4 %1, off" " sc0 sc1" "\n\t"
                 "global_load_lds_dwordx4 %1, off offset:1024" " sc0 sc1" "\n\t"
                 "global_load_lds_dwordx4 %1, off offset:2048" " sc0 sc1" "\n\t"
                 "global_load_lds_dwordx4 %1, off offset:3072" " sc0 sc1" "\n\t"
                 "s_mov_b32 m0, %0"
                 : "=&s"(keep) : "v"(g), "s"(lds) : "memory");
}
__device__ __forceinline__ void st_out(void* p, u32x4 v) {
    *(u32x4*)p = v;
}
#define WAIT_V(n) asm volatile("s_waitcnt vmcnt(" #n ")" ::: "memory")
#define WAIT_L(n) asm volatile("s_waitcnt lgkmcnt(" #n ")" ::: "memory")
#define BAR() do { asm volatile("s_waitcnt lgkmcnt(0)" ::: "memory"); __builtin_amdgcn_s_barrier(); asm volatile("" ::: "memory"); } while (0)

__device__ __forceinline__ u64 ld_sys64(const void* p) {
    return __hip_atomic_load((const u64*)p, __ATOMIC_RELAXED, __HIP_MEMORY_SCOPE_SYSTEM);
}
__device__ __forceinline__ void st_sys64(void* p, u64 v) {
    asm volatile("global_store_dwordx2 %0, %1, off sc0 sc1\n\ts_nop 1" :: "v"(p), "v"(v) : "memory");
}
__device__ __forceinline__ unsigned ld_sys32(const void* p) {
    return __hip_atomic_load((const unsigned*)p, __ATOMIC_RELAXED, __HIP_MEMORY_SCOPE_SYSTEM);
}
__device__ __forceinline__ void st_sys32(void* p, unsigned v) {
    asm volatile("global_store_dword %0, %1, off sc0 sc1\n\ts_nop 1" :: "v"(p), "v"(v) : "memory");
}
__device__ __forceinline__ unsigned ld_ag32(const void* p) {
    return __hip_atomic_load((const unsigned*)p, __ATOMIC_RELAXED, __HIP_MEMORY_SCOPE_AGENT);
}
__device__ __forceinline__ void st_ag32(void* p, unsigned v) {
    __hip_atomic_store((unsigned*)p, v, __ATOMIC_RELAXED, __HIP_MEMORY_SCOPE_AGENT);
}
__device__ __forceinline__ bool expired(char* me, unsigned call, u64 t0) {
    bool e = ld_ag32(me + OFF_TMO) == call || __builtin_amdgcn_s_memrealtime() - t0 > DEADLINE;
    e = __builtin_amdgcn_readfirstlane(e);
    if (e) st_ag32(me + OFF_TMO, call);
    return e;
}
__device__ __forceinline__ void wave_scan6(unsigned c, unsigned& pre, unsigned& tot) {
    pre = 0; tot = 0;
#pragma unroll
    for (int b = 0; b < 6; ++b) {
        u64 m = __builtin_amdgcn_ballot_w64((c >> b) & 1);
        unsigned p = __builtin_amdgcn_mbcnt_hi((unsigned)(m >> 32), __builtin_amdgcn_mbcnt_lo((unsigned)m, 0));
        pre += p << b;
        tot += (unsigned)__builtin_popcountll(m) << b;
    }
}
__device__ __forceinline__ void tile_of(int g, int& i, int& j) {
    int xcd = g & 7, t = g >> 3;
    i = 2 * (t >> 1) + (xcd >> 2);
    j = 2 * (xcd & 3) + (t & 1);
}
#define STAMP(k) do {} while (0)

#define LAG 5
__device__ __forceinline__ void unit_of(int U, bool& isx, int& grp, int& strip, int& ks) {
    bool late; int o;
    if (U < 40 * LAG) { ks = U / 40; o = U - 40 * ks; late = false; }
    else if (U < 40 * LAG + 64 * (8 - LAG)) {
        const int V = U - 40 * LAG, s = V >> 6; o = V & 63;
        late = o < 24;
        if (late) ks = s; else { ks = s + LAG; o -= 24; }
    } else {
        const int V = U - (40 * LAG + 64 * (8 - LAG)), s = V / 24;
        o = V - 24 * s; ks = 8 - LAG + s; late = true;
    }
    isx = late || o < 8;
    if (isx) { grp = o & 1; strip = (late ? 4 : 0) + (o >> 1); }
    else     { grp = ((o >> 3) + o) & 3; strip = o & 7; }
}
__device__ __forceinline__ int link_unit(int L, int& need) {
    const int B = L / 12, o = L - 12 * B;
    bool late; int ks;
    if (B < LAG) { late = false; ks = B; need = 5 * (B + 1); }
    else if (B < 16 - LAG) { const int V = B - LAG; late = !(V & 1); ks = (V >> 1) + (late ? 0 : LAG); need = 5 * LAG + 8 * (V >> 1) + (late ? 3 : 8); }
    else { late = true; ks = B - 8; need = 3 * B + 19; }
    const int st = late ? 4 + o : (o < 4 ? o : 12 + o);
    return st * 64 + ks;
}

__global__ __launch_bounds__(NTHR, 2) void gemm_ar_fused(Params P) {
    extern __shared__ __attribute__((aligned(16))) char smem[];
    const int tid = threadIdx.x, lane = tid & 63;
    const int wv = __builtin_amdgcn_readfirstlane(tid >> 6);
    const int bid = blockIdx.x, b = (bid - (128 - NSEND)) & 255, rank = P.rank;
    char* const me = P.ws[rank];
    const unsigned lds0 = (unsigned)(uintptr_t)(__attribute__((address_space(3))) char*)smem;
    const u64 t0 = __builtin_amdgcn_s_memrealtime();
    STAMP(0);

    unsigned* misc = (unsigned*)(smem + LDS_MISC);
    unsigned cnt = ld_ag32(me + OFF_CNT + 4 * (bid * 8 + wv));
    if (tid == 0) misc[1] = 0;
    BAR();
    const bool pblk = b < NSEND;
    const int cb = pblk ? bid + 128 : bid < 128 ? bid : bid - NSEND;
    if (cb >= DLY0 && !(pblk && wv < WPB))
        while (__builtin_amdgcn_s_memrealtime() - t0 < (unsigned)min(DLYA + DLYS * (cb - DLY0), DLYM)) __builtin_amdgcn_s_sleep(4);
    for (int p = pblk ? wv - WPB : wv; p >= 0 && p < 8; p += pblk ? 8 - WPB : 8) {
        const int pc = cb < NPUSH ? p * NPUSH + cb : cb * 8 + p;
        const int U = pc >> 2, qd = pc & 3;
        bool isx; int grp, strip, ks; unit_of(U, isx, grp, strip, ks);
        f16x8 c[4];
        if (isx) {
            const int kg = (lane & 7) ^ (lane >> 3);
            const float* s = P.x + (size_t)(2048 * grp + 128 * strip + 32 * qd + (lane >> 3)) * KL + 64 * ks + 4 * kg;
            f32x4 f[8];
#pragma unroll
            for (int t = 0; t < 8; ++t) f[t] = *(const f32x4*)(s + (t >> 1) * 8 * KL + (t & 1) * 32);
#pragma unroll
            for (int e = 0; e < 32; ++e) c[e >> 3][e & 7] = (_Float16)f[e >> 2][e & 3];
        } else {
            const float* s = P.w + (size_t)(64 * ks + 8 * qd) * GN + 1024 * grp + 128 * strip + 4 * (lane & 31) + 2 * (lane >> 5);
            f32x2 f[16];
#pragma unroll
            for (int e = 0; e < 16; ++e) f[e] = *(const f32x2*)(s + (size_t)((e & 3) + ((e >> 2) & 1) * 32 + (e >> 3) * 4) * GN);
#pragma unroll
            for (int e = 0; e < 32; ++e) c[e >> 3][e & 7] = (_Float16)f[(e >> 4) * 8 + (e & 7)][(e >> 3) & 1];
        }
        asm volatile("" : "+v"(cnt));
        const unsigned call = cnt + 1;
        const int st = isx ? strip : 16 + strip;
        const size_t li = (size_t)(st * 64 + rank * 8 + ks) * 4 + qd;
        const bool push = cb < NPUSH, mine = grp == (isx ? rank >> 2 : rank & 3);
        if (push) {
#pragma unroll
            for (int h = 0; h < 4; ++h) {
                for (int t = 0; t < (isx ? 4 : 2); ++t) {
                    const unsigned d = (unsigned)(isx ? 4 * grp + ((t + qd) & 3) : grp + 4 * ((t + qd) & 1)) & (NR - 1);
                    st_wt(P.ws[d] + OFF_X + li * 4096 + lane * 16 + h * 1024, __builtin_bit_cast(u32x4, c[h]));
                    if (isx) WAIT_V(3); else WAIT_V(1);
                }
            }
        } else {
            char* s = me + OFF_S + (size_t)((isx ? grp * 16 + strip : 32 + grp * 8 + strip) * 8 + ks) * UNIT + qd * 4096 + lane * 16;
#pragma unroll
            for (int h = 0; h < 4; ++h) st_wt(s + h * 1024, __builtin_bit_cast(u32x4, c[h]));
            if (mine) {
#pragma unroll
                for (int h = 0; h < 4; ++h) st_wt(me + OFF_X + li * 4096 + lane * 16 + h * 1024, __builtin_bit_cast(u32x4, c[h]));
            }
        }
        WAIT_V(0);
        if (push) {
            if (lane < (isx ? 4 : 2)) st_sys32(ws_sel(P, isx ? 4 * grp + lane : grp + 4 * lane) + OFF_LF + li * 4, call);
        } else {
            unsigned k = 0;
            if (lane == 0) k = __hip_atomic_fetch_add(&misc[1], 1u, __ATOMIC_RELAXED, __HIP_MEMORY_SCOPE_WORKGROUP);
            if (__builtin_amdgcn_readfirstlane(k) == 7 && lane < NR && lane != rank)
                st_sys32(ws_sel(P, lane) + OFF_AF + (rank * 256 + cb) * 4, call);
            if (mine && lane == 0) st_sys32(me + OFF_LF + li * 4, call);
        }
        STAMP(1);
    }
    const unsigned call = __builtin_amdgcn_readfirstlane(cnt) + 1;
    if (lane == 0) st_ag32(me + OFF_CNT + 4 * (bid * 8 + wv), call);

    if (b >= NSEND && b < NSEND + NGEMM) {
        const int g = b - NSEND;
        int ti, tj; tile_of(g, ti, tj);
        const size_t fo = (size_t)((rank + lane) & 7) * 128 + (lane >> 3) * 16;
        const char* fw = me + OFF_LF + (size_t)(16 + tj) * 1024 + fo;
        const char* fx = me + OFF_LF + (size_t)ti * 1024 + fo;
        u64 avail = 0, left = ~0ull;
        auto pick = [&](int kk) {
            while (!avail) {
                u32x4 a, c;
                asm volatile("global_load_dwordx4 %0, %2, off sc0 sc1\n\tglobal_load_dwordx4 %1, %3, off sc0 sc1\n\ts_waitcnt vmcnt(0)"
                             : "=&v"(a), "=&v"(c) : "v"(fw), "v"(fx) : "memory");
                const unsigned lo = min(min(min(a.x, a.y), min(a.z, a.w)), min(min(c.x, c.y), min(c.z, c.w)));
                avail = left & __builtin_amdgcn_ballot_w64(lo >= call);
                if (avail) break;
                if (expired(me, call, t0)) { avail = left; break; }
                __builtin_amdgcn_s_sleep(12);
            }
            const int id = __builtin_ctzll(avail);
            avail &= avail - 1; left &= ~(1ull << id);
            if (lane == 0) misc[8 + (kk & 7)] = ((rank + id) & 7) * 8 + (id >> 3);
        };
        if (wv == 0) { pick(0); pick(1); pick(2); }
        BAR();
        STAMP(1);
        const int wr = wv >> 1, wc = wv & 1, q = lane >> 4, r = lane & 15;
        f32x4 acc[2][4];
#pragma unroll
        for (int mi = 0; mi < 2; ++mi)
#pragma unroll
            for (int nn = 0; nn < 4; ++nn) acc[mi][nn] = (f32x4){0.f, 0.f, 0.f, 0.f};
        {
            const char* gsrc = me + OFF_X + (size_t)(wv < 4 ? ti : 16 + tj) * 64 * UNIT
                               + (wv & 3) * 4096 + lane * 16;
            const unsigned ldst = lds0 + wv * 4096;
            auto stage = [&](int kk) {
                glds4(gsrc + (size_t)(misc[8 + (kk & 7)] & 63u) * UNIT, ldst + (kk & 3) * LDS_STAGE);
            };
            stage(0); stage(1); stage(2);
            const int aoff = (32 * wr + r) * 128, boff = 16384 + q * 2048 + (16 * wc + r) * 16;
            for (int kk = 0; kk < 64; ++kk) {
                if (wv == 0 && kk + 3 < 64) pick(kk + 3);
                if (kk < 62) WAIT_V(8); else if (kk == 62) WAIT_V(4); else WAIT_V(0);
                BAR();
                if (kk + 3 < 64) stage(kk + 3);
                const char* st = smem + (kk & 3) * LDS_STAGE;
#pragma unroll
                for (int s = 0; s < 2; ++s) {
                    f16x8 af[2], bf[4];
#pragma unroll
                    for (int mi = 0; mi < 2; ++mi)
                        af[mi] = *(const f16x8*)(st + aoff + mi * 2048 + (((4 * s + q) ^ (r & 7)) * 16));
#pragma unroll
                    for (int nn = 0; nn < 4; ++nn)
                        bf[nn] = *(const f16x8*)(st + boff + s * 8192 + (nn & 1) * 1024 + (nn >> 1) * 512);
#pragma unroll
                    for (int mi = 0; mi < 2; ++mi)
#pragma unroll
                        for (int nn = 0; nn < 4; ++nn)
                            acc[mi][nn] = __builtin_amdgcn_mfma_f32_16x16x32_f16(af[mi], bf[nn], acc[mi][nn], 0, 0, 0);
                }
            }
        }
        BAR();
        if (ld_ag32(me + OFF_TMO) == call) {
#pragma unroll
            for (int mi = 0; mi < 2; ++mi)
#pragma unroll
                for (int nn = 0; nn < 4; ++nn) acc[mi][nn] = (f32x4){NAN, NAN, NAN, NAN};
        }
        STAMP(2);
        unsigned u[32], mask = 0, umax = 0, umin1 = 0xffffffffu;
#pragma unroll
        for (int mi = 0; mi < 2; ++mi)
#pragma unroll
            for (int j = 0; j < 4; ++j)
#pragma unroll
                for (int nn = 0; nn < 4; ++nn) {
                    const int e = (mi * 4 + j) * 4 + nn;
                    const float v = acc[mi][nn][j];
                    const bool keep = !(v <= 0.f);
                    u[e] = keep ? __float_as_uint(v) : 0u;
                    mask |= (keep ? 1u : 0u) << e;
                    umax = max(umax, u[e]);
                    umin1 = min(umin1, u[e] - 1u);
                }
#pragma unroll
        for (int o = 32; o > 0; o >>= 1) {
            umax = max(umax, (unsigned)__shfl_xor((int)umax, o));
            umin1 = min(umin1, (unsigned)__shfl_xor((int)umin1, o));
        }
        unsigned pre, n; wave_scan6(__builtin_popcount(mask), pre, n);
        n = __builtin_amdgcn_readfirstlane(n);
        const unsigned ebase = __builtin_amdgcn_readfirstlane((umin1 + 1u) >> 23);
        const bool raw = __builtin_amdgcn_readfirstlane((umax >> 23) - ebase) >= 32u || ebase > 255u;
        const unsigned sub = raw ? 0u : ebase << 23;
        char* ml = smem + wv * WAVE_LDS;
        *(unsigned*)(ml + lane * 4) = mask;
        {
            unsigned pos = 256 + pre * 4;
#pragma unroll
            for (int e = 0; e < 32; ++e) {
                const bool k = (mask >> e) & 1;
                *(unsigned*)(ml + (k ? pos : 12288u + lane * 4)) = u[e] - sub;
                pos += k ? 4 : 0;
            }
        }
        const unsigned nq = (n + 31) >> 5;
        if (!raw && lane < nq) {
            const u32x4 h = *(const u32x4*)(ml + 256 + (7 * nq + lane) * 16);
#pragma unroll
            for (int p = 0; p < 7; ++p) {
                u32x4* a = (u32x4*)(ml + 256 + (p * nq + lane) * 16);
                *a = (*a & 0x0fffffffu) | ((h >> (4 * p)) << 28);
            }
        }
        const unsigned Q = 16 + (raw ? (n + 3) >> 2 : 7 * nq);
        const int msg = g * 8 + wv;
        {
            char* dst = me + OFF_O + (size_t)msg * SLOT + lane * 16;
            for (unsigned i = 0; i < Q; i += 64)
                if (i + lane < Q) st_wt(dst + i * 16, *(const u32x4*)(ml + (i + lane) * 16));
        }
        STAMP(3);
        unsigned qi = 0;
        if (lane == 0) qi = __hip_atomic_fetch_add((unsigned*)(me + OFF_TICK), 1u, __ATOMIC_RELAXED, __HIP_MEMORY_SCOPE_AGENT);
        qi = __builtin_amdgcn_readfirstlane(qi) & (NMSG - 1);
        WAIT_V(0);
        const u64 fl = ((u64)call << 32) | n | ((ebase & 255u) << 12) | ((unsigned)msg << 20) | (raw ? 0x80000000u : 0u);
        if (lane < NR && lane != rank)
            st_sys64(ws_sel(P, lane) + OFF_Q + (size_t)(rank * NMSG + (qi ^ (STAG * lane))) * 8, fl);
        STAMP(10);
        {
            float* o = P.out + (size_t)(2048 * (rank >> 2) + 128 * ti + 32 * wr + 4 * q) * GN
                       + 1024 * (rank & 3) + 128 * tj + 64 * wc + 4 * r;
#pragma unroll
            for (int e4 = 0; e4 < 8; ++e4)
                st_out(o + (size_t)((e4 >> 2) * 16 + (e4 & 3)) * GN, (u32x4){u[4 * e4], u[4 * e4 + 1], u[4 * e4 + 2], u[4 * e4 + 3]});
        }
        return;
    }
    const int pw = b * WPB + wv, pj = pw / 7;
    if (wv >= WPB || pj >= NP) return;

    const unsigned src = (unsigned)(rank + 1 + pw - 7 * pj) & (NR - 1);
    char* const peer = P.ws[src];
    char* const ml = smem + wv * 2 * SLOT;
    const unsigned mlds = lds0 + wv * 2 * SLOT;
    bool bad = false;
    auto draw = [&](unsigned& t, const char* ctr) {
        if (lane == 0) asm volatile("global_atomic_add %0, %1, %2, off sc0 sc1" : "=v"(t) : "v"(ctr), "v"(1u) : "memory");
    };
    const char* const qrow = me + OFF_Q + (size_t)src * NMSG * 8;
    const unsigned ibase = (call - 1) * (unsigned)(NPIECE - P0 - NPI) - (P0 + 2 * NPI);
    if (pj >= NPI) {
        while (ld_sys32(me + OFF_JT + src * 128) - ibase < (unsigned)NPIECE) {
            if (expired(me, call, t0)) { bad = true; break; }
            __builtin_amdgcn_s_sleep(32);
        }
    } else {
        const char* ctr = me + OFF_JT + src * 128;
        int ok = 0;
        auto fetch = [&](int pi) {
            int need;
            const int lu = link_unit(pi >> 2, need), st = lu >> 6;
            while (ok < need) {
                u32x4 a;
                asm volatile("global_load_dwordx4 %0, %1, off sc0 sc1\n\ts_waitcnt vmcnt(0)"
                             : "=&v"(a) : "v"(me + OFF_AF + src * 1024 + lane * 16) : "memory");
                const u64 m = __builtin_amdgcn_ballot_w64(lane >= NPUSH / 4 && min(min(a.x, a.y), min(a.z, a.w)) < call);
                ok = m ? __builtin_ctzll(m) : 64;
                if (ok >= need) break;
                if (__builtin_amdgcn_s_memrealtime() - t0 > 10000 && expired(me, call, t0)) { bad = true; ok = 64; }
                __builtin_amdgcn_s_sleep(1);
            }
            const unsigned sidx = (unsigned)((st < 16 ? (rank >> 2) * 16 + st : 16 + (rank & 3) * 8 + st) * 8 + (lu & 7)) * 4 + (pi & 3);
            glds4s(peer + OFF_S + (size_t)sidx * 4096 + lane * 16, mlds);
            return (unsigned)(lu + src * 8) * 4 + (pi & 3);
        };
        unsigned fi = fetch(P0 + pj), nxt = P0 + NPI + pj;
        STAMP(3);
        WAIT_V(0);
        STAMP(2);
        while (true) {
            u32x4 v[4];
#pragma unroll
            for (int h = 0; h < 4; ++h) v[h] = *(const u32x4*)(ml + h * 1024 + lane * 16);
            char* d = me + OFF_X + (size_t)fi * 4096 + lane * 16;
#pragma unroll
            for (int h = 0; h < 4; ++h) st_wt(d + h * 1024, v[h]);
            char* fp = me + OFF_LF + fi * 4;
            if (nxt >= (unsigned)NPIECE) {
                WAIT_V(0);
                st_sys32(fp, call);
                break;
            }
            fi = fetch(nxt);
            unsigned t = 0;
            draw(t, ctr);
            WAIT_V(5);
            st_sys32(fp, call);
            asm volatile("s_waitcnt vmcnt(1)" : "+v"(t) :: "memory");
            nxt = __builtin_amdgcn_readfirstlane(t) - ibase;
        }
        STAMP(8);
    }
    {
        const char* ctr = me + OFF_JT + 1024 + src * 128;
        const unsigned base = (call - 1) * (unsigned)(NMSG - 2 * NP) - 3 * NP;
        auto fetch = [&](unsigned info, unsigned p) {
            const unsigned n = min(info & 0xfffu, 2048u);
            const unsigned Q = 16 + ((info >> 31) ? (n + 3) >> 2 : 7 * ((n + 31) >> 5));
            const char* s = peer + OFF_O + (size_t)((info >> 20) & (NMSG - 1)) * SLOT + lane * 16;
            for (unsigned i = 0; i < Q; i += 64)
                if (i + lane < Q) glds1(s + i * 16, mlds + p * SLOT + i * 16);
        };
        auto poll = [&](unsigned j) {
            while (true) {
                const u64 e = ld_sys64(qrow + j * 8);
                if (__builtin_amdgcn_readfirstlane((unsigned)(e >> 32)) >= call) return (unsigned)__builtin_amdgcn_readfirstlane((unsigned)e);
                if (bad || expired(me, call, t0)) { bad = true; return 0u; }
                __builtin_amdgcn_s_sleep(4);
            }
        };
        unsigned jB = NP + pj, jC = 2 * NP + pj, par = 0, eBi = 0;
        bool eBok = false;
        unsigned cur = poll(pj);
        fetch(cur, 0);
        {
            const u64 e = ld_sys64(qrow + jB * 8);
            eBok = __builtin_amdgcn_readfirstlane((unsigned)(e >> 32)) >= call; eBi = __builtin_amdgcn_readfirstlane((unsigned)e);
        }
        WAIT_V(0);
        STAMP(9);
        while (true) {
            const unsigned info = cur;
            char* const m = ml + par * SLOT;
            par ^= 1;
            const bool vB = jB < (unsigned)NMSG, vC = jC < (unsigned)NMSG;
            if (vB) STAMP(10 + (jB >> 8));
            if (vB && eBok) fetch(eBi, par);
            u64 eC = 0; unsigned t = 0;
            if (vC) {
                asm volatile("global_load_dwordx2 %0, %1, off sc0 sc1" : "=v"(eC) : "v"(qrow + jC * 8) : "memory");
                draw(t, ctr);
            }
            const int msg = (info >> 20) & (NMSG - 1);
            int ti, tj; tile_of(msg >> 3, ti, tj);
            float* o = P.out + (size_t)(2048 * (src >> 2) + 128 * ti + 32 * ((msg & 7) >> 1) + 4 * (lane >> 4)) * GN
                       + 1024 * (src & 3) + 128 * tj + 64 * (msg & 1) + 4 * (lane & 15);
            const unsigned n = min(info & 0xfffu, 2048u), nq = (n + 31) >> 5;
            const bool raw = info >> 31;
            const unsigned mask = *(const unsigned*)(m + lane * 4);
            if (!raw && lane < nq) {
                u32x4 h = {0u, 0u, 0u, 0u};
#pragma unroll
                for (int p = 0; p < 7; ++p)
                    h |= (*(const u32x4*)(m + 256 + (p * nq + lane) * 16) >> 28) << (4 * p);
                *(u32x4*)(m + 256 + (7 * nq + lane) * 16) = h;
            }
            unsigned pre, tot; wave_scan6(__builtin_popcount(mask), pre, tot);
            const unsigned am = raw ? 0xffffffffu : 0x0fffffffu, add = raw ? 0u : ((info >> 12) & 0xffu) << 23;
            unsigned pos = 256 + pre * 4;
            u32x4 v[8];
#pragma unroll
            for (int e = 0; e < 32; ++e) {
                v[e >> 2][e & 3] = *(const unsigned*)(m + pos);
                pos += ((mask >> e) & 1u) * 4u;
            }
            asm volatile("" ::: "memory");
#pragma unroll
            for (int e4 = 0; e4 < 8; ++e4) {
#pragma unroll
                for (int c = 0; c < 4; ++c)
                    v[e4][c] = ((v[e4][c] & am) + add) & (unsigned)(((int)(mask << (31 - (4 * e4 + c)))) >> 31);
                asm volatile("global_store_dwordx4 %0, %1, off" OSC "\n\ts_nop 1" :: "v"(o + (size_t)((e4 >> 2) * 16 + (e4 & 3)) * GN), "v"(v[e4]) : "memory");
            }
            asm volatile("s_waitcnt vmcnt(8)" : "+v"(eC), "+v"(t) :: "memory");
            if (!vB) break;
            if (!eBok) { eBi = poll(jB); fetch(eBi, par); WAIT_V(0); }
            cur = eBi;
            jB = jC;
            eBok = __builtin_amdgcn_readfirstlane((unsigned)(eC >> 32)) >= call; eBi = __builtin_amdgcn_readfirstlane((unsigned)eC);
            jC = vC ? __builtin_amdgcn_readfirstlane(t) - base : ~0u;
        }
        if (bad) { WAIT_V(0); P.out[(size_t)(2048 * (src >> 2) + 2047) * GN + 1024 * (src & 3) + lane] = NAN; }
        STAMP(15);
    }
}

extern "C" void kernel_launch(int rank, int nranks, ncclComm_t comm,
                              hipStream_t stream,
                              void* const* d_in, const int* in_sizes, int n_in,
                              void* d_out, int out_size,
                              const int* in_dtypes, int out_dtype,
                              void* const* d_out_all,
                              void* const* d_ws_all, size_t ws_bytes) {
    if (nranks != NR || (unsigned)rank >= (unsigned)NR || ws_bytes < WS_NEED || n_in < 2 ||
        in_sizes[0] != 4096 * KL || in_sizes[1] != KL * GN || out_size != 4096 * GN) {
        fprintf(stderr, "unexpected problem shape\n");
        return;
    }
    HIP_CHECK(hipFuncSetAttribute((const void*)gemm_ar_fused, hipFuncAttributeMaxDynamicSharedMemorySize, LDS_BYTES));
    Params P;
    P.x = (const float*)d_in[0];
    P.w = (const float*)d_in[1];
    P.out = (float*)d_out;
    for (int p = 0; p < NR; ++p) P.ws[p] = (char*)d_ws_all[p];
    P.rank = rank;
    gemm_ar_fused<<<NWG, NTHR, LDS_BYTES, stream>>>(P);
    HIP_CHECK(hipGetLastError());
}
```

```cpp
#include <hip/hip_runtime.h>
#include <rccl/rccl.h>
#include <cstdio>
#include <cstdint>

typedef unsigned u32x4 __attribute__((ext_vector_type(4)));
typedef unsigned u32x2 __attribute__((ext_vector_type(2)));
typedef unsigned short u16x2 __attribute__((ext_vector_type(2)));
typedef float f32x4 __attribute__((ext_vector_type(4)));
typedef float f32x2 __attribute__((ext_vector_type(2)));
typedef _Float16 f16x8 __attribute__((ext_vector_type(8)));
typedef unsigned long long u64;

#define HIP_CHECK(expr) do { hipError_t _e = (expr); if (_e != hipSuccess) { \
    fprintf(stderr, "HIP error %s at %s:%d\n", hipGetErrorString(_e), __FILE__, __LINE__); \
    return; } } while (0)

constexpr int NR = 8;
constexpr int GN = 4096;
constexpr int KL = 512;
constexpr int NWG = 256, NTHR = 512;
#define NP_ 32
#define NPI_ 28
#define DLY0 ((PUSHB + 1) * (16 + NE))
#define DLYA 150
#define DLYS 4
#define DLYM 600
#define PSL 8
#define PHOLD 450
#define WPB 4
constexpr int NP = NP_;
constexpr int NPI = NPI_;
constexpr int NSEND = (7 * NP + WPB - 1) / WPB;
constexpr int NGEMM = 128;
constexpr int NMSG = NGEMM * 8;
constexpr int NPIECE = 768;
constexpr size_t UNIT = 16384;
constexpr size_t SLOT = 9216;
constexpr unsigned long long DEADLINE = 40000000ull;
#define PUSHB 1
#define NE 8
constexpr int NPUSH = (16 + NE) * PUSHB;
constexpr int P0 = 4 * (8 + NE) * PUSHB;
static_assert(NSEND + NPUSH <= 128, "");

constexpr size_t OFF_CNT   = 57344;
constexpr size_t OFF_TMO   = 4096;
constexpr size_t OFF_TICK  = 6144;
constexpr size_t OFF_LF    = 16384;
constexpr size_t OFF_AF    = 49152;
constexpr size_t OFF_JT    = 8192;
constexpr size_t OFF_DBG   = 262144;
constexpr size_t OFF_Q     = 65536;
constexpr size_t OFF_X     = 1u << 20;
constexpr size_t OFF_S     = OFF_X + 24 * 64 * UNIT;
constexpr size_t OFF_O     = OFF_S + 512 * UNIT;
constexpr size_t OFF_R     = OFF_O + (size_t)NMSG * SLOT;
constexpr size_t WS_NEED   = OFF_R + 512 * UNIT;

constexpr int LDS_STAGE = 32768, LDS_MISC = 4 * LDS_STAGE, LDS_SNAP = LDS_MISC + 1024, LDS_BYTES = LDS_SNAP + 2048;
constexpr int WAVE_LDS = 16384;

struct Params { const float* x; const float* w; float* out; char* ws[NR]; int rank; };

__device__ __forceinline__ char* ws_sel(const Params& P, int d) {
    char* r = P.ws[0];
#pragma unroll
    for (int p = 1; p < NR; ++p) { char* w = P.ws[p]; asm("" : "+s"(w)); r = d == p ? w : r; }
    return r;
}
__device__ __forceinline__ void st_wt(void* p, u32x4 v) {
    asm volatile("global_store_dwordx4 %0, %1, off sc0 sc1\n\ts_nop 1" :: "v"(p), "v"(v) : "memory");
}
__device__ __forceinline__ void glds1(const void* g, unsigned lds) {
    unsigned keep;
    asm volatile("s_mov_b32 %0, m0\n\ts_mov_b32 m0, %2\n\ts_nop 0\n\t"
                 "global_load_lds_dwordx4 %1, off sc0 sc1\n\ts_mov_b32 m0, %0"
                 : "=&s"(keep) : "v"(g), "s"(lds) : "memory");
}
__device__ __forceinline__ void glds4(const void* g, unsigned lds) {
    unsigned keep;
    asm volatile("s_mov_b32 %0, m0\n\ts_mov_b32 m0, %2\n\ts_nop 0\n\t"
                 "global_load_lds_dwordx4 %1, off sc0 sc1\n\t"
                 "global_load_lds_dwordx4 %1, off offset:1024 sc0 sc1\n\t"
                 "global_load_lds_dwordx4 %1, off offset:2048 sc0 sc1\n\t"
                 "global_load_lds_dwordx4 %1, off offset:3072 sc0 sc1\n\t"
                 "s_mov_b32 m0, %0"
                 : "=&s"(keep) : "v"(g), "s"(lds) : "memory");
}
#define WAIT_V(n) asm volatile("s_waitcnt vmcnt(" #n ")" ::: "memory")
#define WAIT_L(n) asm volatile("s_waitcnt lgkmcnt(" #n ")" ::: "memory")
#define BAR() do { asm volatile("s_waitcnt lgkmcnt(0)" ::: "memory"); __builtin_amdgcn_s_barrier(); asm volatile("" ::: "memory"); } while (0)

__device__ __forceinline__ u64 ld_sys64(const void* p) {
    return __hip_atomic_load((const u64*)p, __ATOMIC_RELAXED, __HIP_MEMORY_SCOPE_SYSTEM);
}
__device__ __forceinline__ void st_sys64(void* p, u64 v) {
    asm volatile("global_store_dwordx2 %0, %1, off sc0 sc1\n\ts_nop 1" :: "v"(p), "v"(v) : "memory");
}
__device__ __forceinline__ unsigned ld_sys32(const void* p) {
    return __hip_atomic_load((const unsigned*)p, __ATOMIC_RELAXED, __HIP_MEMORY_SCOPE_SYSTEM);
}
__device__ __forceinline__ void st_sys32(void* p, unsigned v) {
    asm volatile("global_store_dword %0, %1, off sc0 sc1\n\ts_nop 1" :: "v"(p), "v"(v) : "memory");
}
__device__ __forceinline__ unsigned ld_ag32(const void* p) {
    return __hip_atomic_load((const unsigned*)p, __ATOMIC_RELAXED, __HIP_MEMORY_SCOPE_AGENT);
}
__device__ __forceinline__ void st_ag32(void* p, unsigned v) {
    __hip_atomic_store((unsigned*)p, v, __ATOMIC_RELAXED, __HIP_MEMORY_SCOPE_AGENT);
}
__device__ __forceinline__ bool expired(char* me, unsigned call, u64 t0) {
    const u64 dt = __builtin_amdgcn_s_memrealtime() - t0;
    if (dt < 100000) return false;
    bool e = ld_ag32(me + OFF_TMO) == call || dt > DEADLINE;
    e = __builtin_amdgcn_readfirstlane(e);
    if (e) st_ag32(me + OFF_TMO, call);
    return e;
}
__device__ __forceinline__ void wave_scan6(unsigned c, unsigned& pre, unsigned& tot) {
    pre = 0; tot = 0;
#pragma unroll
    for (int b = 0; b < 6; ++b) {
        u64 m = __builtin_amdgcn_ballot_w64((c >> b) & 1);
        unsigned p = __builtin_amdgcn_mbcnt_hi((unsigned)(m >> 32), __builtin_amdgcn_mbcnt_lo((unsigned)m, 0));
        pre += p << b;
        tot += (unsigned)__builtin_popcountll(m) << b;
    }
}
__device__ __forceinline__ void tile_of(int g, int& i, int& j) {
    int xcd = g & 7, t = g >> 3;
    i = 2 * (t >> 1) + (xcd >> 2);
    j = 2 * (xcd & 3) + (t & 1);
}
#define STAMP(k) do {} while (0)

#define LAG 7
template <int NA, int NL> __device__ __forceinline__ void seq_of(int U, bool& late, int& ks, int& o) {
    if (U < NA * LAG) { ks = U / NA; o = U - NA * ks; late = false; }
    else if (U < NA * LAG + (NA + NL) * (8 - LAG)) {
        const int V = U - NA * LAG, s = V / (NA + NL); o = V - s * (NA + NL);
        late = o < NL;
        if (late) ks = s; else { ks = s + LAG; o -= NL; }
    } else {
        const int V = U - (NA * LAG + (NA + NL) * (8 - LAG)), s = V / NL;
        o = V - NL * s; ks = 8 - LAG + s; late = true;
    }
}
__device__ __forceinline__ void unit_of(int U, bool& isx, int& grp, int& strip, int& ks) {
    bool late; int o; seq_of<32 + 2 * NE, 32 - 2 * NE>(U, late, ks, o);
    isx = late || o < 2 * NE;
    if (isx) { grp = o & 1; strip = (late ? NE : 0) + (o >> 1); }
    else     { o -= 2 * NE; grp = ((o >> 3) + o) & 3; strip = o & 7; }
}
__device__ __forceinline__ unsigned msg_quads(unsigned info) {
    const unsigned L = min(info & 127u, 64u), ne = (info >> 7) & 127u;
    return 16 + (ne > 64u ? 8 * L : (104 * L + 2 * ne + 15) >> 4);
}
__device__ __forceinline__ int link_unit(int L, int rank, int& U) {
    bool late; int ks, o; seq_of<8 + NE, 16 - NE>(L, late, ks, o);
    const int na = late ? min(ks + LAG, 8) : ks, nl = late ? ks : max(ks + 1 - LAG, 0);
    U = na * (32 + 2 * NE) + nl * (32 - 2 * NE)
        + (late || o < NE ? 2 * o + (rank >> 2) : NE + o + 8 * ((rank - o + NE) & 3));
    return (late ? NE + o : o < NE ? o : 16 - NE + o) * 64 + ks;
}

__global__ __launch_bounds__(NTHR, 2) void gemm_ar_fused(Params P) {
    extern __shared__ __attribute__((aligned(16))) char smem[];
    const int tid = threadIdx.x, lane = tid & 63;
    const int wv = __builtin_amdgcn_readfirstlane(tid >> 6);
    const int bid = blockIdx.x, b = (bid - (128 - NSEND)) & 255, rank = P.rank;
    char* const me = P.ws[rank];
    const unsigned lds0 = (unsigned)(uintptr_t)(__attribute__((address_space(3))) char*)smem;
    const u64 t0 = __builtin_amdgcn_s_memrealtime();
    STAMP(0);

    unsigned* misc = (unsigned*)(smem + LDS_MISC);
    unsigned cnt = ld_ag32(me + OFF_CNT + 4 * (bid * 8 + wv));
    if (tid == 0) { misc[1] = 0; misc[2] = 0; }
    if (tid < 128) *(u32x4*)(smem + LDS_SNAP + tid * 16) = (u32x4){0u, 0u, 0u, 0u};
    BAR();
    const bool pblk = b < NSEND;
    const int cb = pblk ? bid + 128 : bid < 128 ? bid : bid - NSEND;
    if (cb >= DLY0 && !(pblk && wv < WPB))
        while (__builtin_amdgcn_s_memrealtime() - t0 < (unsigned)min(DLYA + DLYS * (cb - DLY0), DLYM)) __builtin_amdgcn_s_sleep(4);
    for (int p = pblk ? wv - WPB : wv; p >= 0 && p < 8; p += pblk ? 8 - WPB : 8) {
        const int pc = cb < NPUSH ? p * NPUSH + cb : cb * 8 + p;
        const int U = pc >> 2, qd = pc & 3;
        bool isx; int grp, strip, ks; unit_of(U, isx, grp, strip, ks);
        f16x8 c[4];
        if (isx) {
            const int kg = (lane & 7) ^ (lane >> 3);
            const float* s = P.x + (size_t)(2048 * grp + 128 * strip + 32 * qd + (lane >> 3)) * KL + 64 * ks + 4 * kg;
            f32x4 f[8];
#pragma unroll
            for (int t = 0; t < 8; ++t) f[t] = *(const f32x4*)(s + (t >> 1) * 8 * KL + (t & 1) * 32);
#pragma unroll
            for (int e = 0; e < 32; ++e) c[e >> 3][e & 7] = (_Float16)f[e >> 2][e & 3];
        } else {
            const float* s = P.w + (size_t)(64 * ks + 8 * qd) * GN + 1024 * grp + 128 * strip + 4 * (lane & 31) + 2 * (lane >> 5);
            f32x2 f[16];
#pragma unroll
            for (int e = 0; e < 16; ++e) f[e] = *(const f32x2*)(s + (size_t)((e & 3) + ((e >> 2) & 1) * 32 + (e >> 3) * 4) * GN);
#pragma unroll
            for (int e = 0; e < 32; ++e) c[e >> 3][e & 7] = (_Float16)f[(e >> 4) * 8 + (e & 7)][(e >> 3) & 1];
        }
        asm volatile("" : "+v"(cnt));
        const unsigned call = cnt + 1;
        const int st = isx ? strip : 16 + strip;
        const size_t li = (size_t)(st * 64 + rank * 8 + ks) * 4 + qd;
        const bool push = cb < NPUSH, mine = grp == (isx ? rank >> 2 : rank & 3);
        if (push) {
#pragma unroll
            for (int h = 0; h < 4; ++h) {
                for (int t = 0; t < (isx ? 4 : 2); ++t) {
                    const unsigned d = (unsigned)(isx ? 4 * grp + ((t + qd) & 3) : grp + 4 * ((t + qd) & 1)) & (NR - 1);
                    st_wt(P.ws[d] + OFF_X + li * 4096 + lane * 16 + h * 1024, __builtin_bit_cast(u32x4, c[h]));
                }
            }
        } else {
            const size_t so = (size_t)(((isx ? grp * 16 + strip : 32 + grp * 8 + strip) * 8 + ks) * 4 + qd) * 4096;
            unsigned d[16], T[2] = {0u, 0u};
            u16x2 mx = {0, 0};
#pragma unroll
            for (int j = 0; j < 16; ++j) {
                d[j] = __builtin_bit_cast(u32x4, c[j >> 2])[j & 3];
                mx = __builtin_elementwise_max(mx, __builtin_bit_cast(u16x2, d[j] & 0x7fff7fffu));
            }
            unsigned em = max(mx.x, mx.y);
#pragma unroll
            for (int o = 32; o > 0; o >>= 1) em = max(em, (unsigned)__shfl_xor((int)em, o));
            const unsigned low = max((unsigned)__builtin_amdgcn_readfirstlane(em) >> 10, 7u) - 7u;
#pragma unroll
            for (int j = 0; j < 16; ++j) {
                d[j] = __builtin_bit_cast(unsigned, __builtin_bit_cast(u16x2, d[j]) - __builtin_bit_cast(u16x2, low * 0x04000400u));
                T[j >> 3] |= ((d[j] >> 13) & 0x00030003u) << (2 * (j & 7));
            }
            u64 z = ((u64)T[1] << 32) | T[0];
            u64 nz = (z | (z >> 1)) & 0x5555555555555555ull;
            unsigned epre, nesc; wave_scan6(__builtin_popcountll(nz), epre, nesc);
            char* el = smem + LDS_MISC - 2048 + wv * 256;
            if (nesc > 63u) {
#pragma unroll
                for (int h = 0; h < 4; ++h) st_wt(me + OFF_R + so + h * 1024 + lane * 16, __builtin_bit_cast(u32x4, c[h]));
            } else {
                for (; nz; nz &= nz - 1) {
                    const int bit = __builtin_ctzll(nz);
                    *(unsigned short*)(el + 2 * ++epre) = (lane << 7) | (bit << 1) | ((unsigned)(z >> bit) & 3u);
                }
#pragma unroll
                for (int j = 0; j < 14; ++j) {
                    const unsigned g = d[14 + j / 7], p = j % 7;
                    const unsigned top = p < 6 ? g << (13 - 2 * p) : ((g << 1) & 0x20002000u) | ((g >> 1) & 0x40004000u);
                    d[j] = (d[j] & 0x9fff9fffu) | (top & 0x60006000u);
                }
#pragma unroll
                for (int h = 0; h < 3; ++h)
                    st_wt(me + OFF_S + so + h * 1024 + lane * 16, (u32x4){d[4 * h], d[4 * h + 1], d[4 * h + 2], d[4 * h + 3]});
                st_sys64(me + OFF_S + so + 3072 + lane * 8, ((u64)d[13] << 32) | d[12]);
            }
            if (lane == 0) *(unsigned short*)el = min(nesc, 64u) | (low << 8);
            asm volatile("" ::: "memory");
            if (lane < 8) st_wt(me + OFF_S + so + 3584 + lane * 16, *(const u32x4*)(el + lane * 16));
            if (mine) {
#pragma unroll
                for (int h = 0; h < 4; ++h) st_wt(me + OFF_X + li * 4096 + lane * 16 + h * 1024, __builtin_bit_cast(u32x4, c[h]));
            }
        }
        WAIT_V(0);
        if (push) {
            if (lane < (isx ? 4 : 2)) st_sys32(ws_sel(P, isx ? 4 * grp + lane : grp + 4 * lane) + OFF_LF + li * 4, call);
        } else {
            unsigned k = 0;
            if (lane == 0) k = __hip_atomic_fetch_add(&misc[1], 1u, __ATOMIC_RELAXED, __HIP_MEMORY_SCOPE_WORKGROUP);
            if (__builtin_amdgcn_readfirstlane(k) == 7 && lane < NR && lane != rank)
                st_sys32(ws_sel(P, lane) + OFF_AF + (rank * 256 + cb) * 4, call);
            if (mine && lane == 0) st_sys32(me + OFF_LF + li * 4, call);
        }
        STAMP(1);
    }
    const unsigned call = __builtin_amdgcn_readfirstlane(cnt) + 1;
    if (lane == 0) st_ag32(me + OFF_CNT + 4 * (bid * 8 + wv), call);

    if (bid >= 128) {
        const int g = bid - 128;
        int ti, tj; tile_of(g, ti, tj);
        const size_t fo = (size_t)((rank + lane) & 7) * 128 + (lane >> 3) * 16;
        const char* fw = me + OFF_LF + (size_t)(16 + tj) * 1024;
        const char* fx = me + OFF_LF + (size_t)ti * 1024;
        u64 avail = 0, left = ~0ull;
        constexpr int nst = 64;
        auto arrived = [&](u32x4 a, u32x4 c) {
            return left & __builtin_amdgcn_ballot_w64(min(min(min(a.x, a.y), min(a.z, a.w)), min(min(c.x, c.y), min(c.z, c.w))) >= call);
        };
        auto pick = [&](int kk) {
            if (!avail) avail = arrived(*(const u32x4*)(smem + LDS_SNAP + fo), *(const u32x4*)(smem + LDS_SNAP + 1024 + fo));
            while (!avail) {
                u32x4 a, c;
                asm volatile("global_load_dwordx4 %0, %2, off sc0 sc1\n\tglobal_load_dwordx4 %1, %3, off sc0 sc1\n\ts_waitcnt vmcnt(0)"
                             : "=&v"(a), "=&v"(c) : "v"(fw + fo), "v"(fx + fo) : "memory");
                avail = arrived(a, c);
                if (avail) break;
                if (expired(me, call, t0)) { avail = left; misc[2] = 1; break; }
                __builtin_amdgcn_s_sleep(PSL);
            }
            const int id = __builtin_ctzll(avail);
            avail &= avail - 1; left &= ~(1ull << id);
            if (lane == 0) misc[8 + (kk & 7)] = ((rank + id) & 7) * 8 + (id >> 3);
        };
        STAMP(1);
        const int kh = wv >> 2, wr = (wv & 2) + kh, wc = wv & 1, q = lane >> 4, r = lane & 15;
        f32x4 acc[2][4];
        {
            f32x4 ac[4][4];
#pragma unroll
            for (int mi = 0; mi < 4; ++mi)
#pragma unroll
                for (int nn = 0; nn < 4; ++nn) ac[mi][nn] = (f32x4){0.f, 0.f, 0.f, 0.f};
            const char* gsrc = me + OFF_X + (size_t)(wv < 4 ? ti : 16 + tj) * 64 * UNIT
                               + (wv & 3) * 4096 + lane * 16;
            const unsigned ldst = lds0 + wv * 4096;
            auto stage = [&](int kk) {
                if (wv == 0) glds1(((kk & 1) ? fw : fx) + lane * 16, lds0 + LDS_SNAP + (kk & 1) * 1024);
                glds4(gsrc + (size_t)(misc[8 + (kk & 7)] & 63u) * UNIT, ldst + (kk & 3) * LDS_STAGE);
            };
            for (int kk = 0; kk < 3; ++kk) {
                if (wv == 0) pick(kk);
                BAR();
                stage(kk);
            }
            const int aoff = (32 * (wv & 2) + r) * 128 + ((4 * kh + q) ^ (r & 7)) * 16;
            const int boff = 16384 + kh * 8192 + q * 2048 + (16 * wc + r) * 16;
            for (int kk = 0; kk < nst; ++kk) {
                if (wv == 0 && kk + 3 < nst) pick(kk + 3);
                if (kk + 2 < nst) { if (wv == 0) WAIT_V(10); else WAIT_V(8); }
                else if (kk + 1 < nst) WAIT_V(4); else WAIT_V(0);
                BAR();
                if (kk + 3 < nst) stage(kk + 3);
                const char* st = smem + (kk & 3) * LDS_STAGE;
                f16x8 af[4], bf[4];
#pragma unroll
                for (int i = 0; i < 4; ++i) {
                    af[i] = *(const f16x8*)(st + aoff + i * 2048);
                    bf[i] = *(const f16x8*)(st + boff + (i & 1) * 1024 + (i >> 1) * 512);
                }
#pragma unroll
                for (int mi = 0; mi < 4; ++mi)
#pragma unroll
                    for (int nn = 0; nn < 4; ++nn)
                        ac[mi][nn] = __builtin_amdgcn_mfma_f32_16x16x32_f16(af[mi], bf[nn], ac[mi][nn], 0, 0, 0);
            }
            BAR();
#pragma unroll
            for (int i = 0; i < 8; ++i)
                *(f32x4*)(smem + (wv ^ 4) * WAVE_LDS + i * 1024 + lane * 16) = kh ? ac[i >> 2][i & 3] : ac[2 + (i >> 2)][i & 3];
            BAR();
#pragma unroll
            for (int i = 0; i < 8; ++i)
                acc[i >> 2][i & 3] = (kh ? ac[2 + (i >> 2)][i & 3] : ac[i >> 2][i & 3])
                                     + *(const f32x4*)(smem + wv * WAVE_LDS + i * 1024 + lane * 16);
        }
        const bool poison = misc[2];
        unsigned qi = 0;
        if (lane == 0) asm volatile("global_atomic_add %0, %1, %2, off sc0 sc1" : "=v"(qi) : "v"(me + OFF_TICK), "v"(1u) : "memory");
        if (poison) {
#pragma unroll
            for (int i = 0; i < 8; ++i) acc[i >> 2][i & 3] = (f32x4){NAN, NAN, NAN, NAN};
        }
        STAMP(2);
        unsigned u[32], mask = 0, umax = 0;
#pragma unroll
        for (int mi = 0; mi < 2; ++mi)
#pragma unroll
            for (int j = 0; j < 4; ++j)
#pragma unroll
                for (int nn = 0; nn < 4; ++nn) {
                    const int e = (mi * 4 + j) * 4 + nn;
                    const float v = acc[mi][nn][j];
                    const bool keep = !(v <= 0.f);
                    u[e] = keep ? __float_as_uint(v) : 0u;
                    mask |= (keep ? 1u : 0u) << e;
                    umax = max(umax, u[e]);
                }
#pragma unroll
        for (int o = 32; o > 0; o >>= 1) umax = max(umax, (unsigned)__shfl_xor((int)umax, o));
        umax = __builtin_amdgcn_readfirstlane(umax);
        const unsigned ebase = max(umax >> 23, 7u) - 7u;
        unsigned em = 0;
#pragma unroll
        for (int e = 0; e < 32; ++e) em |= (u[e] < (ebase << 23) ? 1u : 0u) << e;
        em &= mask;
        unsigned pre, n, epre, nesc;
        wave_scan6(__builtin_popcount(mask), pre, n);
        wave_scan6(__builtin_popcount(em), epre, nesc);
        const bool raw = nesc > 64u || (umax >> 31);
        const unsigned sub = raw ? 0u : ebase << 23;
        char* ml = smem + wv * WAVE_LDS;
        *(unsigned*)(ml + lane * 4) = mask;
        {
            unsigned pos = 256 + pre * 4;
#pragma unroll
            for (int e = 0; e < 32; ++e) {
                const bool k = (mask >> e) & 1;
                *(unsigned*)(ml + (k ? pos : 12288u + lane * 4)) = u[e] - sub;
                pos += k ? 4 : 0;
            }
        }
        const unsigned L = (n + 31) >> 5;
        if (!raw) {
            for (; em; em &= em - 1) {
                const unsigned pos = pre + __builtin_popcount(mask & ((em & -em) - 1u));
                *(unsigned short*)(ml + 13312 + 2 * epre++) = pos | ((*(const unsigned*)(ml + 256 + 4 * pos) >> 26) << 11);
            }
            if (lane < L) {
                char* pl = ml + 256 + lane * 8;
                u32x2 h[3];
#pragma unroll
                for (int k = 0; k < 3; ++k) h[k] = *(const u32x2*)(pl + (13 + k) * 8 * L);
#pragma unroll
                for (int p = 0; p < 13; ++p) {
                    u32x2* a = (u32x2*)(pl + p * 8 * L);
                    const u32x2 top = p < 12 ? h[p >> 2] << (26 - 6 * (p & 3))
                                             : ((h[0] << 2) & 0x0c000000u) | ((h[1] << 4) & 0x30000000u) | ((h[2] << 6) & 0xc0000000u);
                    *a = (*a & 0x03ffffffu) | (top & 0xfc000000u);
                }
            }
            asm volatile("" ::: "memory");
            *(unsigned short*)(ml + 256 + 104 * L + 2 * lane) = *(const unsigned short*)(ml + 13312 + 2 * lane);
        }
        const unsigned info = L | ((raw ? 127u : nesc) << 7) | ((ebase & 255u) << 14) | ((unsigned)(g * 8 + wr * 2 + wc) << 22);
        const unsigned Q = msg_quads(info);
        const int msg = g * 8 + wr * 2 + wc;
        {
            char* dst = me + OFF_O + (size_t)msg * SLOT + lane * 16;
            for (unsigned i = 0; i < Q; i += 64)
                if (i + lane < Q) st_wt(dst + i * 16, *(const u32x4*)(ml + (i + lane) * 16));
        }
        STAMP(3);
        asm volatile("s_waitcnt vmcnt(0)" : "+v"(qi) :: "memory");
        qi = __builtin_amdgcn_readfirstlane(qi) & (NMSG - 1);
        const u64 fl = ((u64)call << 32) | info;
        if (lane < NR && lane != rank)
            st_sys64(ws_sel(P, lane) + OFF_Q + (size_t)(rank * NMSG + qi) * 8, fl);
        STAMP(10);
        {
            float* o = P.out + (size_t)(2048 * (rank >> 2) + 128 * ti + 32 * wr + 4 * q) * GN
                       + 1024 * (rank & 3) + 128 * tj + 64 * wc + 4 * r;
#pragma unroll
            for (int e4 = 0; e4 < 8; ++e4)
                *(u32x4*)(o + (size_t)((e4 >> 2) * 16 + (e4 & 3)) * GN) = (u32x4){u[4 * e4], u[4 * e4 + 1], u[4 * e4 + 2], u[4 * e4 + 3]};
        }
        return;
    }
    const int pw = b * WPB + wv, pj = pw / 7;
    if (wv >= WPB || pj >= NP) return;

    const unsigned src = (unsigned)(rank + 1 + pw - 7 * pj) & (NR - 1);
    char* const peer = P.ws[src];
    char* const ml = smem + wv * 2 * SLOT;
    const unsigned mlds = lds0 + wv * 2 * SLOT;
    bool bad = false;
    auto draw = [&](unsigned& t, const char* ctr) {
        if (lane == 0) asm volatile("global_atomic_add %0, %1, %2, off sc0 sc1" : "=v"(t) : "v"(ctr), "v"(1u) : "memory");
    };
    const char* const qrow = me + OFF_Q + (size_t)src * NMSG * 8;
    const unsigned ibase = (call - 1) * (unsigned)(NPIECE - P0 - NPI) - (P0 + 2 * NPI);
    if (pj >= NPI) {
        while (ld_sys32(me + OFF_JT + src * 128) - ibase < (unsigned)NPIECE) {
            if (expired(me, call, t0)) { bad = true; break; }
            __builtin_amdgcn_s_sleep(32);
        }
    } else {
        const char* ctr = me + OFF_JT + src * 128;
        u64 ok = 0;
        while (__builtin_amdgcn_s_memrealtime() - t0 < PHOLD) __builtin_amdgcn_s_sleep(2);
        auto fetch = [&](int pi, unsigned p) {
            int U;
            const int lu = link_unit(pi >> 2, rank, U), st = lu >> 6;
            while (!((ok >> (U >> 3)) & 1)) {
                u32x4 a;
                asm volatile("global_load_dwordx4 %0, %1, off sc0 sc1\n\ts_waitcnt vmcnt(0)"
                             : "=&v"(a) : "v"(me + OFF_AF + src * 1024 + lane * 16) : "memory");
                ok = __builtin_amdgcn_ballot_w64(min(min(a.x, a.y), min(a.z, a.w)) >= call);
                if ((ok >> (U >> 3)) & 1) break;
                if (expired(me, call, t0)) { bad = true; ok = ~0ull; }
                __builtin_amdgcn_s_sleep(1);
            }
            const unsigned sidx = (unsigned)((st < 16 ? (rank >> 2) * 16 + st : 16 + (rank & 3) * 8 + st) * 8 + (lu & 7)) * 4 + (pi & 3);
            const char* s = peer + OFF_S + (size_t)sidx * 4096 + lane * 16;
            glds1(s, mlds + p * SLOT); glds1(s + 1024, mlds + p * SLOT + 1024); glds1(s + 2048, mlds + p * SLOT + 2048);
            if (lane < 40) glds1(s + 3072, mlds + p * SLOT + 3072);
            return (((unsigned)(lu + src * 8) * 4 + (pi & 3)) << 16) | sidx;
        };
        unsigned cur = fetch(P0 + pj, 0), nxt = P0 + NPI + pj, par = 0;
        char* fp = me + 256;
        STAMP(3);
        WAIT_V(0);
        STAMP(2);
        while (true) {
            char* const m = ml + par * SLOT;
            const unsigned id = cur;
            const bool more = nxt < (unsigned)NPIECE;
            unsigned t = 0;
            par ^= 1;
            if (more) { cur = fetch(nxt, par); draw(t, ctr); }
            const unsigned esc = *(const unsigned short*)(m + 3584 + 2 * lane);
            *(u64*)(m + 4096 + lane * 8) = 0;
            u32x4 v[4];
#pragma unroll
            for (int h = 0; h < 3; ++h) v[h] = *(const u32x4*)(m + h * 1024 + lane * 16);
            v[3].xy = *(const u32x2*)(m + 3072 + lane * 8);
            const unsigned hdr = __builtin_amdgcn_readfirstlane(esc);
            if (hdr & 64u) {
                WAIT_L(0);
                glds4(peer + OFF_R + (size_t)(id & 2047u) * 4096 + lane * 16, mlds + (par ^ 1) * SLOT);
                WAIT_V(0);
#pragma unroll
                for (int h = 0; h < 4; ++h) v[h] = *(const u32x4*)(m + h * 1024 + lane * 16);
            } else {
                if (lane - 1u < (hdr & 63u))
                    __hip_atomic_fetch_or((unsigned*)(m + 4096 + ((esc >> 6) & 127u) * 4), (esc & 3u) << ((esc >> 1) & 30u), __ATOMIC_RELAXED, __HIP_MEMORY_SCOPE_WAVEFRONT);
                asm volatile("" ::: "memory");
                const u32x2 T = *(const u32x2*)(m + 4096 + lane * 8);
#pragma unroll
                for (int k = 0; k < 2; ++k) {
                    const unsigned g = v[(7 * k + 6) >> 2][(7 * k + 6) & 3];
                    unsigned r = ((g >> 1) & 0x10001000u) | ((g << 1) & 0x80008000u);
#pragma unroll
                    for (int p = 0; p < 6; ++p) r |= (v[(7 * k + p) >> 2][(7 * k + p) & 3] >> (13 - 2 * p)) & (0x00030003u << (2 * p));
                    v[3][2 + k] = r;
                }
                const u16x2 add = __builtin_bit_cast(u16x2, (hdr >> 8) * 0x04000400u);
#pragma unroll
                for (int j = 0; j < 16; ++j) {
                    const unsigned up = (j & 7) < 7 ? T[j >> 3] << (13 - 2 * (j & 7)) : T[j >> 3] >> 1;
                    v[j >> 2][j & 3] = __builtin_bit_cast(unsigned,
                        __builtin_bit_cast(u16x2, (v[j >> 2][j & 3] & 0x9fff9fffu) | (up & 0x60006000u)) + add);
                }
            }
            if (more) WAIT_V(5); else WAIT_V(0);
            st_sys32(fp, call);
            char* d = me + OFF_X + (size_t)(id >> 16) * 4096 + lane * 16;
#pragma unroll
            for (int h = 0; h < 4; ++h) st_wt(d + h * 1024, v[h]);
            fp = me + OFF_LF + (id >> 16) * 4;
            if (!more) break;
            asm volatile("s_waitcnt vmcnt(5)" : "+v"(t) :: "memory");
            nxt = __builtin_amdgcn_readfirstlane(t) - ibase;
        }
        WAIT_V(0);
        st_sys32(fp, call);
        STAMP(8);
    }
    {
        const char* ctr = me + OFF_JT + 1024 + src * 128;
        const unsigned base = (call - 1) * (unsigned)(NMSG - 2 * NP) - 3 * NP;
        auto fetch = [&](unsigned info, unsigned p) {
            const unsigned Q = msg_quads(info);
            const char* s = peer + OFF_O + (size_t)(info >> 22) * SLOT + lane * 16;
            for (unsigned i = 0; i < Q; i += 64)
                if (i + lane < Q) glds1(s + i * 16, mlds + p * SLOT + i * 16);
        };
        auto poll = [&](unsigned j) {
            while (true) {
                const u64 e = ld_sys64(qrow + j * 8);
                if (__builtin_amdgcn_readfirstlane((unsigned)(e >> 32)) >= call) return (unsigned)__builtin_amdgcn_readfirstlane((unsigned)e);
                if (bad || expired(me, call, t0)) { bad = true; return 0u; }
                __builtin_amdgcn_s_sleep(4);
            }
        };
        unsigned jB = NP + pj, jC = 2 * NP + pj, par = 0, eBi = 0;
        bool eBok = false;
        unsigned cur = poll(pj);
        fetch(cur, 0);
        {
            const u64 e = ld_sys64(qrow + jB * 8);
            eBok = __builtin_amdgcn_readfirstlane((unsigned)(e >> 32)) >= call; eBi = __builtin_amdgcn_readfirstlane((unsigned)e);
        }
        WAIT_V(0);
        STAMP(9);
        while (true) {
            const unsigned info = cur;
            char* const m = ml + par * SLOT;
            par ^= 1;
            const bool vB = jB < (unsigned)NMSG, vC = jC < (unsigned)NMSG;
            if (vB) STAMP(10 + (jB >> 8));
            if (vB && eBok) fetch(eBi, par);
            u64 eC = 0; unsigned t = 0;
            if (vC) {
                asm volatile("global_load_dwordx2 %0, %1, off sc0 sc1" : "=v"(eC) : "v"(qrow + jC * 8) : "memory");
                draw(t, ctr);
            }
            const int msg = info >> 22;
            int ti, tj; tile_of(msg >> 3, ti, tj);
            float* o = P.out + (size_t)(2048 * (src >> 2) + 128 * ti + 32 * ((msg & 7) >> 1) + 4 * (lane >> 4)) * GN
                       + 1024 * (src & 3) + 128 * tj + 64 * (msg & 1) + 4 * (lane & 15);
            const unsigned L = min(info & 127u, 64u), ne = (info >> 7) & 127u;
            const unsigned mask = *(const unsigned*)(m + lane * 4);
            if (ne <= 64u) {
                const unsigned esc = *(const unsigned short*)(m + 256 + 104 * L + 2 * lane), add = ((info >> 14) & 255u) << 23;
                asm volatile("" ::: "memory");
                if (lane < L) {
                    char* pl = m + 256 + lane * 8;
                    u32x2 w[13], h[3];
#pragma unroll
                    for (int p = 0; p < 13; ++p) w[p] = *(const u32x2*)(pl + p * 8 * L);
#pragma unroll
                    for (int k = 0; k < 3; ++k) {
                        h[k] = (w[12] >> (2 + 2 * k)) & 0x03000000u;
#pragma unroll
                        for (int c = 0; c < 4; ++c) h[k] |= (w[4 * k + c] >> 26) << (6 * c);
                        *(u32x2*)(pl + (13 + k) * 8 * L) = h[k] + add;
                    }
#pragma unroll
                    for (int p = 0; p < 13; ++p) *(u32x2*)(pl + p * 8 * L) = (w[p] & 0x03ffffffu) + add;
                }
                if (lane < ne)
                    __hip_atomic_fetch_add((unsigned*)(m + 256 + 4 * (esc & 2047u)), (32u | (esc >> 11)) << 26, __ATOMIC_RELAXED, __HIP_MEMORY_SCOPE_WAVEFRONT);
                asm volatile("" ::: "memory");
            }
            unsigned pre, tot; wave_scan6(__builtin_popcount(mask), pre, tot);
            unsigned pos = 256 + pre * 4;
            u32x4 v[8];
#pragma unroll
            for (int e = 0; e < 32; ++e) {
                v[e >> 2][e & 3] = *(const unsigned*)(m + pos);
                pos += ((mask >> e) & 1u) * 4u;
            }
            asm volatile("" ::: "memory");
#pragma unroll
            for (int e4 = 0; e4 < 8; ++e4) {
#pragma unroll
                for (int c = 0; c < 4; ++c)
                    v[e4][c] &= (unsigned)(((int)(mask << (31 - (4 * e4 + c)))) >> 31);
                asm volatile("global_store_dwordx4 %0, %1, off\n\ts_nop 1" :: "v"(o + (size_t)((e4 >> 2) * 16 + (e4 & 3)) * GN), "v"(v[e4]) : "memory");
            }
            asm volatile("s_waitcnt vmcnt(8)" : "+v"(eC), "+v"(t) :: "memory");
            if (!vB) break;
            if (!eBok) { eBi = poll(jB); fetch(eBi, par); WAIT_V(0); }
            cur = eBi;
            jB = jC;
            eBok = __builtin_amdgcn_readfirstlane((unsigned)(eC >> 32)) >= call; eBi = __builtin_amdgcn_readfirstlane((unsigned)eC);
            jC = vC ? __builtin_amdgcn_readfirstlane(t) - base : ~0u;
        }
        if (bad) { WAIT_V(0); P.out[(size_t)(2048 * (src >> 2) + 2047) * GN + 1024 * (src & 3) + lane] = NAN; }
        STAMP(15);
    }
}

extern "C" void kernel_launch(int rank, int nranks, ncclComm_t comm,
                              hipStream_t stream,
                              void* const* d_in, const int* in_sizes, int n_in,
                              void* d_out, int out_size,
                              const int* in_dtypes, int out_dtype,
                              void* const* d_out_all,
                              void* const* d_ws_all, size_t ws_bytes) {
    if (nranks != NR || (unsigned)rank >= (unsigned)NR || ws_bytes < WS_NEED || n_in < 2 ||
        in_sizes[0] != 4096 * KL || in_sizes[1] != KL * GN || out_size != 4096 * GN) {
        fprintf(stderr, "unexpected problem shape\n");
        return;
    }
    HIP_CHECK(hipFuncSetAttribute((const void*)gemm_ar_fused, hipFuncAttributeMaxDynamicSharedMemorySize, LDS_BYTES));
    Params P;
    P.x = (const float*)d_in[0];
    P.w = (const float*)d_in[1];
    P.out = (float*)d_out;
    for (int p = 0; p < NR; ++p) P.ws[p] = (char*)d_ws_all[p];
    P.rank = rank;
    gemm_ar_fused<<<NWG, NTHR, LDS_BYTES, stream>>>(P);
    HIP_CHECK(hipGetLastError());
}
```

```cpp
#include <hip/hip_runtime.h>
#include <rccl/rccl.h>
#include <cstdio>
#include <cstdint>

typedef unsigned u32x4 __attribute__((ext_vector_type(4)));
typedef unsigned u32x2 __attribute__((ext_vector_type(2)));
typedef unsigned short u16x2 __attribute__((ext_vector_type(2)));
typedef float f32x4 __attribute__((ext_vector_type(4)));
typedef float f32x2 __attribute__((ext_vector_type(2)));
typedef _Float16 f16x8 __attribute__((ext_vector_type(8)));
typedef unsigned long long u64;

#define HIP_CHECK(expr) do { hipError_t _e = (expr); if (_e != hipSuccess) { \
    fprintf(stderr, "HIP error %s at %s:%d\n", hipGetErrorString(_e), __FILE__, __LINE__); \
    return; } } while (0)

constexpr int NR = 8;
constexpr int GN = 4096;
constexpr int KL = 512;
constexpr int NWG = 256, NTHR = 512;
#define NP_ 28
#define NPI_ 26
#define DLY0 ((PUSHB + 1) * (16 + NE))
#define DLYA 150
#define DLYS 4
#define DLYM 600
#define PSL 16
#define PHOLD 300
#define AFS 1
#define WPB 2
constexpr int NP = NP_;
constexpr int NPI = NPI_;
constexpr int NSEND = 112;
constexpr int NGEMM = 128;
constexpr int NMSG = NGEMM * 8;
constexpr int NPIECE = 768;
constexpr size_t UNIT = 16384;
constexpr size_t SLOT = 9216;
constexpr unsigned long long DEADLINE = 40000000ull;
#define PUSHB 1
#define NE 8
constexpr int NPUSH = (16 + NE) * PUSHB;
constexpr int P0 = 4 * (8 + NE) * PUSHB;


constexpr size_t OFF_CNT   = 57344;
constexpr size_t OFF_TMO   = 4096;
constexpr size_t OFF_TICK  = 6144;
constexpr size_t OFF_LF    = 16384;
constexpr size_t OFF_AF    = 49152;
constexpr size_t OFF_JT    = 8192;
constexpr size_t OFF_Q     = 65536;
constexpr size_t OFF_X     = 1u << 20;
constexpr size_t OFF_S     = OFF_X + 24 * 64 * UNIT;
constexpr size_t OFF_O     = OFF_S + 512 * UNIT;
constexpr size_t OFF_R     = OFF_O + (size_t)NMSG * SLOT;
constexpr size_t WS_NEED   = OFF_R + 512 * UNIT;

constexpr int LDS_STAGE = 32768, LDS_MISC = 4 * LDS_STAGE, LDS_SNAP = LDS_MISC + 1024, LDS_BYTES = LDS_SNAP + 2048;
constexpr int WAVE_LDS = 16384;

struct Params { const float* x; const float* w; float* out; char* ws[NR]; int rank; };

__device__ __forceinline__ char* ws_sel(const Params& P, int d) {
    char* r = P.ws[0];
#pragma unroll
    for (int p = 1; p < NR; ++p) { char* w = P.ws[p]; asm("" : "+s"(w)); r = d == p ? w : r; }
    return r;
}
__device__ __forceinline__ void st_wt(void* p, u32x4 v) {
    asm volatile("global_store_dwordx4 %0, %1, off sc0 sc1\n\ts_nop 1" :: "v"(p), "v"(v) : "memory");
}
__device__ __forceinline__ void glds1(const void* g, unsigned lds) {
    unsigned keep;
    asm volatile("s_mov_b32 %0, m0\n\ts_mov_b32 m0, %2\n\ts_nop 0\n\t"
                 "global_load_lds_dwordx4 %1, off sc0 sc1\n\ts_mov_b32 m0, %0"
                 : "=&s"(keep) : "v"(g), "s"(lds) : "memory");
}
__device__ __forceinline__ void glds4(const void* g, unsigned lds) {
    unsigned keep;
    asm volatile("s_mov_b32 %0, m0\n\ts_mov_b32 m0, %2\n\ts_nop 0\n\t"
                 "global_load_lds_dwordx4 %1, off sc0 sc1\n\t"
                 "global_load_lds_dwordx4 %1, off offset:1024 sc0 sc1\n\t"
                 "global_load_lds_dwordx4 %1, off offset:2048 sc0 sc1\n\t"
                 "global_load_lds_dwordx4 %1, off offset:3072 sc0 sc1\n\t"
                 "s_mov_b32 m0, %0"
                 : "=&s"(keep) : "v"(g), "s"(lds) : "memory");
}
#define WAIT_V(n) asm volatile("s_waitcnt vmcnt(" #n ")" ::: "memory")
#define WAIT_L(n) asm volatile("s_waitcnt lgkmcnt(" #n ")" ::: "memory")
#define BAR() do { asm volatile("s_waitcnt lgkmcnt(0)" ::: "memory"); __builtin_amdgcn_s_barrier(); asm volatile("" ::: "memory"); } while (0)

__device__ __forceinline__ u64 ld_sys64(const void* p) {
    return __hip_atomic_load((const u64*)p, __ATOMIC_RELAXED, __HIP_MEMORY_SCOPE_SYSTEM);
}
__device__ __forceinline__ void st_sys64(void* p, u64 v) {
    asm volatile("global_store_dwordx2 %0, %1, off sc0 sc1\n\ts_nop 1" :: "v"(p), "v"(v) : "memory");
}
__device__ __forceinline__ unsigned ld_sys32(const void* p) {
    return __hip_atomic_load((const unsigned*)p, __ATOMIC_RELAXED, __HIP_MEMORY_SCOPE_SYSTEM);
}
__device__ __forceinline__ void st_sys32(void* p, unsigned v) {
    asm volatile("global_store_dword %0, %1, off sc0 sc1\n\ts_nop 1" :: "v"(p), "v"(v) : "memory");
}
__device__ __forceinline__ unsigned ld_ag32(const void* p) {
    return __hip_atomic_load((const unsigned*)p, __ATOMIC_RELAXED, __HIP_MEMORY_SCOPE_AGENT);
}
__device__ __forceinline__ void st_ag32(void* p, unsigned v) {
    __hip_atomic_store((unsigned*)p, v, __ATOMIC_RELAXED, __HIP_MEMORY_SCOPE_AGENT);
}
__device__ __forceinline__ bool expired(char* me, unsigned call, u64 t0) {
    const u64 dt = __builtin_amdgcn_s_memrealtime() - t0;
    if (dt < 100000) return false;
    bool e = ld_ag32(me + OFF_TMO) == call || dt > DEADLINE;
    e = __builtin_amdgcn_readfirstlane(e);
    if (e) st_ag32(me + OFF_TMO, call);
    return e;
}
__device__ __forceinline__ void wave_scan6(unsigned c, unsigned& pre, unsigned& tot) {
    pre = 0; tot = 0;
#pragma unroll
    for (int b = 0; b < 6; ++b) {
        u64 m = __builtin_amdgcn_ballot_w64((c >> b) & 1);
        unsigned p = __builtin_amdgcn_mbcnt_hi((unsigned)(m >> 32), __builtin_amdgcn_mbcnt_lo((unsigned)m, 0));
        pre += p << b;
        tot += (unsigned)__builtin_popcountll(m) << b;
    }
}
__device__ __forceinline__ void tile_of(int g, int& i, int& j) {
    int xcd = g & 7, t = g >> 3;
    i = 2 * (t >> 1) + (xcd >> 2);
    j = 2 * (xcd & 3) + (t & 1);
}

#define LAG 8
template <int NA, int NL> __device__ __forceinline__ void seq_of(int U, bool& late, int& ks, int& o) {
    if (U < NA * LAG) { ks = U / NA; o = U - NA * ks; late = false; }
    else if (U < NA * LAG + (NA + NL) * (8 - LAG)) {
        const int V = U - NA * LAG, s = V / (NA + NL); o = V - s * (NA + NL);
        late = o < NL;
        if (late) ks = s; else { ks = s + LAG; o -= NL; }
    } else {
        const int V = U - (NA * LAG + (NA + NL) * (8 - LAG)), s = V / NL;
        o = V - NL * s; ks = 8 - LAG + s; late = true;
    }
}
__device__ __forceinline__ void unit_of(int U, bool& isx, int& grp, int& strip, int& ks) {
    bool late; int o; seq_of<32 + 2 * NE, 32 - 2 * NE>(U, late, ks, o);
    isx = late || o < 2 * NE;
    if (isx) { grp = o & 1; strip = (late ? NE : 0) + (o >> 1); }
    else     { o -= 2 * NE; grp = ((o >> 3) + o) & 3; strip = o & 7; }
}
__device__ __forceinline__ unsigned msg_quads(unsigned info) {
    const unsigned n = min(info & 4095u, 2048u), ne = (info >> 12) & 127u;
    return 16 + (ne > 63u ? (n + 3) >> 2 : (104 * (n >> 5) + 4 * (n & 31u) + 2 * ne + 17) >> 4);
}
__device__ __forceinline__ int link_unit(int L, int rank, int& U) {
    bool late; int ks, o; seq_of<8 + NE, 16 - NE>(L, late, ks, o);
    const int na = late ? min(ks + LAG, 8) : ks, nl = late ? ks : max(ks + 1 - LAG, 0);
    U = na * (32 + 2 * NE) + nl * (32 - 2 * NE)
        + (late || o < NE ? 2 * o + (rank >> 2) : NE + o + 8 * ((rank - o + NE) & 3));
    return (late ? NE + o : o < NE ? o : 16 - NE + o) * 64 + ks;
}

__global__ __launch_bounds__(NTHR, 2) void gemm_ar_fused(Params P) {
    extern __shared__ __attribute__((aligned(16))) char smem[];
    const int tid = threadIdx.x, lane = tid & 63;
    const int wv = __builtin_amdgcn_readfirstlane(tid >> 6);
    const int bid = blockIdx.x, b = (bid - (128 - NSEND)) & 255, rank = P.rank;
    char* const me = P.ws[rank];
    const unsigned lds0 = (unsigned)(uintptr_t)(__attribute__((address_space(3))) char*)smem;
    const u64 t0 = __builtin_amdgcn_s_memrealtime();

    unsigned* misc = (unsigned*)(smem + LDS_MISC);
    unsigned cnt = ld_ag32(me + OFF_CNT + 4 * (bid * 8 + wv));
    if (tid == 0) { misc[1] = 0; misc[2] = 0; }
    if (tid < 128) *(u32x4*)(smem + LDS_SNAP + tid * 16) = (u32x4){0u, 0u, 0u, 0u};
    BAR();
    const bool pblk = b < NSEND;
    const int cb = pblk ? bid + 128 : bid < 128 ? bid : bid >= 248 ? bid - 232 : bid - 104;
    if (cb >= DLY0 && !(pblk && wv < WPB))
        while (__builtin_amdgcn_s_memrealtime() - t0 < (unsigned)min(DLYA + DLYS * (cb - DLY0), DLYM)) __builtin_amdgcn_s_sleep(4);
    for (int p = pblk ? wv - WPB : wv; p >= 0 && p < 8; p += pblk ? 8 - WPB : 8) {
        const int pc = cb < NPUSH ? p * NPUSH + cb : cb * 8 + p;
        const int U = pc >> 2, qd = pc & 3;
        bool isx; int grp, strip, ks; unit_of(U, isx, grp, strip, ks);
        f16x8 c[4];
        if (isx) {
            const int kg = (lane & 7) ^ (lane >> 3);
            const float* s = P.x + (size_t)(2048 * grp + 128 * strip + 32 * qd + (lane >> 3)) * KL + 64 * ks + 4 * kg;
            f32x4 f[8];
#pragma unroll
            for (int t = 0; t < 8; ++t) f[t] = *(const f32x4*)(s + (t >> 1) * 8 * KL + (t & 1) * 32);
#pragma unroll
            for (int e = 0; e < 32; ++e) c[e >> 3][e & 7] = (_Float16)f[e >> 2][e & 3];
        } else {
            const float* s = P.w + (size_t)(64 * ks + 8 * qd) * GN + 1024 * grp + 128 * strip + 4 * (lane & 31) + 2 * (lane >> 5);
            f32x2 f[16];
#pragma unroll
            for (int e = 0; e < 16; ++e) f[e] = *(const f32x2*)(s + (size_t)((e & 3) + ((e >> 2) & 1) * 32 + (e >> 3) * 4) * GN);
#pragma unroll
            for (int e = 0; e < 32; ++e) c[e >> 3][e & 7] = (_Float16)f[(e >> 4) * 8 + (e & 7)][(e >> 3) & 1];
        }
        asm volatile("" : "+v"(cnt));
        const unsigned call = cnt + 1;
        const int st = isx ? strip : 16 + strip;
        const size_t li = (size_t)(st * 64 + rank * 8 + ks) * 4 + qd;
        const bool push = cb < NPUSH;
        const bool mine = grp == (isx ? rank >> 2 : rank & 3);
        if (push) {
#pragma unroll
            for (int h = 0; h < 4; ++h) {
                for (int t = 0; t < (isx ? 4 : 2); ++t) {
                    const unsigned d = (unsigned)(isx ? 4 * grp + ((t + qd) & 3) : grp + 4 * ((t + qd) & 1)) & (NR - 1);
                    st_wt(P.ws[d] + OFF_X + li * 4096 + lane * 16 + h * 1024, __builtin_bit_cast(u32x4, c[h]));
                }
            }
        } else {
            const size_t so = (size_t)(((isx ? grp * 16 + strip : 32 + grp * 8 + strip) * 8 + ks) * 4 + qd) * 4096;
            unsigned d[16], T[2] = {0u, 0u};
            u16x2 mx = {0, 0};
#pragma unroll
            for (int j = 0; j < 16; ++j) {
                d[j] = __builtin_bit_cast(u32x4, c[j >> 2])[j & 3];
                mx = __builtin_elementwise_max(mx, __builtin_bit_cast(u16x2, d[j] & 0x7fff7fffu));
            }
            unsigned em = max(mx.x, mx.y);
#pragma unroll
            for (int o = 32; o > 0; o >>= 1) em = max(em, (unsigned)__shfl_xor((int)em, o));
            const unsigned low = max((unsigned)__builtin_amdgcn_readfirstlane(em) >> 10, 7u) - 7u;
#pragma unroll
            for (int j = 0; j < 16; ++j) {
                d[j] = __builtin_bit_cast(unsigned, __builtin_bit_cast(u16x2, d[j]) - __builtin_bit_cast(u16x2, low * 0x04000400u));
                T[j >> 3] |= ((d[j] >> 13) & 0x00030003u) << (2 * (j & 7));
            }
            u64 z = ((u64)T[1] << 32) | T[0];
            u64 nz = (z | (z >> 1)) & 0x5555555555555555ull;
            unsigned epre, nesc; wave_scan6(__builtin_popcountll(nz), epre, nesc);
            char* el = smem + LDS_MISC - 2048 + wv * 256;
            if (nesc > 63u) {
#pragma unroll
                for (int h = 0; h < 4; ++h) st_wt(me + OFF_R + so + h * 1024 + lane * 16, __builtin_bit_cast(u32x4, c[h]));
            } else {
                for (; nz; nz &= nz - 1) {
                    const int bit = __builtin_ctzll(nz);
                    *(unsigned short*)(el + 2 * ++epre) = (lane << 7) | (bit << 1) | ((unsigned)(z >> bit) & 3u);
                }
#pragma unroll
                for (int j = 0; j < 14; ++j) {
                    const unsigned g = d[14 + j / 7], p = j % 7;
                    const unsigned top = p < 6 ? g << (13 - 2 * p) : ((g << 1) & 0x20002000u) | ((g >> 1) & 0x40004000u);
                    d[j] = (d[j] & 0x9fff9fffu) | (top & 0x60006000u);
                }
#pragma unroll
                for (int h = 0; h < 3; ++h)
                    st_wt(me + OFF_S + so + h * 1024 + lane * 16, (u32x4){d[4 * h], d[4 * h + 1], d[4 * h + 2], d[4 * h + 3]});
                st_sys64(me + OFF_S + so + 3072 + lane * 8, ((u64)d[13] << 32) | d[12]);
            }
            if (lane == 0) *(unsigned short*)el = min(nesc, 64u) | (low << 8);
            asm volatile("" ::: "memory");
            if (lane < 8) st_wt(me + OFF_S + so + 3584 + lane * 16, *(const u32x4*)(el + lane * 16));
            if (mine) {
#pragma unroll
                for (int h = 0; h < 4; ++h) st_wt(me + OFF_X + li * 4096 + lane * 16 + h * 1024, __builtin_bit_cast(u32x4, c[h]));
            }
        }
        WAIT_V(0);
        if (push) {
            if (lane < (isx ? 4 : 2)) st_sys32(ws_sel(P, isx ? 4 * grp + lane : grp + 4 * lane) + OFF_LF + li * 4, call);
        } else {
            unsigned k = 0;
            if (lane == 0) k = __hip_atomic_fetch_add(&misc[1], 1u, __ATOMIC_RELAXED, __HIP_MEMORY_SCOPE_WORKGROUP);
            if (__builtin_amdgcn_readfirstlane(k) == 7 && lane < NR && lane != rank)
                st_sys32(ws_sel(P, lane) + OFF_AF + (rank * 256 + cb) * 4, call);
            if (mine && lane == 0) st_sys32(me + OFF_LF + li * 4, call);
        }
    }
    const unsigned call = __builtin_amdgcn_readfirstlane(cnt) + 1;
    if (lane == 0) st_ag32(me + OFF_CNT + 4 * (bid * 8 + wv), call);

    if (bid >= 128) {
        const int g = bid - 128;
        int ti, tj; tile_of(g, ti, tj);
        const size_t fo = (size_t)((rank + lane) & 7) * 128 + (lane >> 3) * 16;
        const char* fw = me + OFF_LF + (size_t)(16 + tj) * 1024;
        const char* fx = me + OFF_LF + (size_t)ti * 1024;
        u64 avail = 0, left = ~0ull;
        constexpr int nst = 64;
        auto arrived = [&](u32x4 a, u32x4 c) {
            return left & __builtin_amdgcn_ballot_w64(min(min(min(a.x, a.y), min(a.z, a.w)), min(min(c.x, c.y), min(c.z, c.w))) >= call);
        };
        auto pick = [&](int kk) {
            if (!avail) avail = arrived(*(const u32x4*)(smem + LDS_SNAP + fo), *(const u32x4*)(smem + LDS_SNAP + 1024 + fo));
            while (!avail) {
                u32x4 a, c;
                asm volatile("global_load_dwordx4 %0, %2, off sc0 sc1\n\tglobal_load_dwordx4 %1, %3, off sc0 sc1\n\ts_waitcnt vmcnt(0)"
                             : "=&v"(a), "=&v"(c) : "v"(fw + fo), "v"(fx + fo) : "memory");
                avail = arrived(a, c);
                if (avail) break;
                if (expired(me, call, t0)) { avail = left; misc[2] = 1; break; }
                __builtin_amdgcn_s_sleep(PSL);
            }
            const int id = __builtin_ctzll(avail);
            avail &= avail - 1; left &= ~(1ull << id);
            if (lane == 0) misc[8 + (kk & 7)] = ((rank + id) & 7) * 8 + (id >> 3);
        };
        const int kh = wv >> 2, wr = (wv & 2) + kh, wc = wv & 1, q = lane >> 4, r = lane & 15;
        f32x4 acc[2][4];
        {
            f32x4 ac[4][4];
#pragma unroll
            for (int mi = 0; mi < 4; ++mi)
#pragma unroll
                for (int nn = 0; nn < 4; ++nn) ac[mi][nn] = (f32x4){0.f, 0.f, 0.f, 0.f};
            const char* gsrc = me + OFF_X + (size_t)(wv < 4 ? ti : 16 + tj) * 64 * UNIT
                               + (wv & 3) * 4096 + lane * 16;
            const unsigned ldst = lds0 + wv * 4096;
            auto stage = [&](int kk) {
                if (wv == 0) glds1(((kk & 1) ? fw : fx) + lane * 16, lds0 + LDS_SNAP + (kk & 1) * 1024);
                glds4(gsrc + (size_t)(misc[8 + (kk & 7)] & 63u) * UNIT, ldst + (kk & 3) * LDS_STAGE);
            };
            for (int kk = 0; kk < 3; ++kk) {
                if (wv == 0) pick(kk);
                BAR();
                stage(kk);
            }
            const int aoff = (32 * (wv & 2) + r) * 128 + ((4 * kh + q) ^ (r & 7)) * 16;
            const int boff = 16384 + kh * 8192 + q * 2048 + (16 * wc + r) * 16;
            for (int kk = 0; kk < nst; ++kk) {
                if (wv == 0 && kk + 3 < nst) pick(kk + 3);
                if (kk + 2 < nst) { if (wv == 0) WAIT_V(10); else WAIT_V(8); }
                else if (kk + 1 < nst) WAIT_V(4); else WAIT_V(0);
                BAR();
                if (kk + 3 < nst) stage(kk + 3);
                const char* st = smem + (kk & 3) * LDS_STAGE;
                f16x8 af[4], bf[4];
#pragma unroll
                for (int i = 0; i < 4; ++i) {
                    af[i] = *(const f16x8*)(st + aoff + i * 2048);
                    bf[i] = *(const f16x8*)(st + boff + (i & 1) * 1024 + (i >> 1) * 512);
                }
#pragma unroll
                for (int mi = 0; mi < 4; ++mi)
#pragma unroll
                    for (int nn = 0; nn < 4; ++nn)
                        ac[mi][nn] = __builtin_amdgcn_mfma_f32_16x16x32_f16(af[mi], bf[nn], ac[mi][nn], 0, 0, 0);
            }
            BAR();
#pragma unroll
            for (int i = 0; i < 8; ++i)
                *(f32x4*)(smem + (wv ^ 4) * WAVE_LDS + i * 1024 + lane * 16) = kh ? ac[i >> 2][i & 3] : ac[2 + (i >> 2)][i & 3];
            BAR();
#pragma unroll
            for (int i = 0; i < 8; ++i)
                acc[i >> 2][i & 3] = (kh ? ac[2 + (i >> 2)][i & 3] : ac[i >> 2][i & 3])
                                     + *(const f32x4*)(smem + wv * WAVE_LDS + i * 1024 + lane * 16);
        }
        const bool poison = misc[2];
        unsigned qi = 0;
        if (lane == 0) asm volatile("global_atomic_add %0, %1, %2, off sc0 sc1" : "=v"(qi) : "v"(me + OFF_TICK), "v"(1u) : "memory");
        if (poison) {
#pragma unroll
            for (int i = 0; i < 8; ++i) acc[i >> 2][i & 3] = (f32x4){NAN, NAN, NAN, NAN};
        }
        unsigned u[32], mask = 0, umax = 0;
#pragma unroll
        for (int mi = 0; mi < 2; ++mi)
#pragma unroll
            for (int j = 0; j < 4; ++j)
#pragma unroll
                for (int nn = 0; nn < 4; ++nn) {
                    const int e = (mi * 4 + j) * 4 + nn;
                    const float v = acc[mi][nn][j];
                    const bool keep = !(v <= 0.f);
                    u[e] = keep ? __float_as_uint(v) : 0u;
                    mask |= (keep ? 1u : 0u) << e;
                    umax = max(umax, u[e]);
                }
#pragma unroll
        for (int o = 32; o > 0; o >>= 1) umax = max(umax, (unsigned)__shfl_xor((int)umax, o));
        umax = __builtin_amdgcn_readfirstlane(umax);
        const unsigned ebase = max(umax >> 23, 7u) - 7u;
        unsigned em = 0;
#pragma unroll
        for (int e = 0; e < 32; ++e) em |= (u[e] < (ebase << 23) ? 1u : 0u) << e;
        em &= mask;
        unsigned pre, n, epre, nesc;
        wave_scan6(__builtin_popcount(mask), pre, n);
        wave_scan6(__builtin_popcount(em), epre, nesc);
        const bool raw = nesc > 63u || (umax >> 31);
        const unsigned sub = raw ? 0u : ebase << 23;
        char* ml = smem + wv * WAVE_LDS;
        *(unsigned*)(ml + lane * 4) = mask;
        {
            unsigned pos = 256 + pre * 4;
#pragma unroll
            for (int e = 0; e < 32; ++e) {
                const bool k = (mask >> e) & 1;
                *(unsigned*)(ml + (k ? pos : 12288u + lane * 4)) = u[e] - sub;
                pos += k ? 4 : 0;
            }
        }
        if (!raw) {
            const unsigned L = n >> 5;
            if (lane == 0) *(unsigned short*)(ml + 13312) = ebase;
            for (; em; em &= em - 1) {
                const unsigned pos = pre + __builtin_popcount(mask & ((em & -em) - 1u));
                *(unsigned short*)(ml + 13312 + 2 * ++epre) = pos | ((*(const unsigned*)(ml + 256 + 4 * pos) >> 26) << 11);
            }
            if (lane < L) {
                char* pl = ml + 256 + lane * 8;
                u32x2 h[3];
#pragma unroll
                for (int k = 0; k < 3; ++k) h[k] = *(const u32x2*)(pl + (13 + k) * 8 * L);
#pragma unroll
                for (int p = 0; p < 13; ++p) {
                    u32x2* a = (u32x2*)(pl + p * 8 * L);
                    const u32x2 top = p < 12 ? h[p >> 2] << (26 - 6 * (p & 3))
                                             : ((h[0] << 2) & 0x0c000000u) | ((h[1] << 4) & 0x30000000u) | ((h[2] << 6) & 0xc0000000u);
                    *a = (*a & 0x03ffffffu) | (top & 0xfc000000u);
                }
            }
            asm volatile("" ::: "memory");
            *(unsigned*)(ml + 256 + 104 * L + 4 * (lane & 31)) = *(const unsigned*)(ml + 256 + 128 * L + 4 * (lane & 31));
            asm volatile("" ::: "memory");
            *(unsigned short*)(ml + 256 + 104 * L + 4 * (n & 31u) + 2 * lane) = *(const unsigned short*)(ml + 13312 + 2 * lane);
        }
        const unsigned info = n | ((raw ? 64u : nesc) << 12) | ((unsigned)(g * 8 + wr * 2 + wc) << 22);
        const unsigned Q = msg_quads(info);
        const int msg = g * 8 + wr * 2 + wc;
        {
            char* dst = me + OFF_O + (size_t)msg * SLOT + lane * 16;
            for (unsigned i = 0; i < Q; i += 64)
                if (i + lane < Q) st_wt(dst + i * 16, *(const u32x4*)(ml + (i + lane) * 16));
        }
        asm volatile("s_waitcnt vmcnt(0)" : "+v"(qi) :: "memory");
        qi = __builtin_amdgcn_readfirstlane(qi) & (NMSG - 1);
        const u64 fl = ((u64)call << 32) | info;
        if (lane < NR && lane != rank)
            st_sys64(ws_sel(P, lane) + OFF_Q + (size_t)(rank * NMSG + qi) * 8, fl);
        {
            float* o = P.out + (size_t)(2048 * (rank >> 2) + 128 * ti + 32 * wr + 4 * q) * GN
                       + 1024 * (rank & 3) + 128 * tj + 64 * wc + 4 * r;
#pragma unroll
            for (int e4 = 0; e4 < 8; ++e4)
                st_wt(o + (size_t)((e4 >> 2) * 16 + (e4 & 3)) * GN, (u32x4){u[4 * e4], u[4 * e4 + 1], u[4 * e4 + 2], u[4 * e4 + 3]});
        }
        return;
    }
    const int pj = b / 7 + 16 * wv, pw = 7 * pj + (b + wv) % 7;
    if (!pblk) return;
    if (wv >= WPB || pj >= NP) return;

    const unsigned src = (unsigned)(rank + 1 + pw - 7 * pj) & (NR - 1);
    char* const peer = P.ws[src];
    char* const ml = smem + wv * 2 * SLOT;
    const unsigned mlds = lds0 + wv * 2 * SLOT;
    bool bad = false;
    auto draw = [&](unsigned& t, const char* ctr) {
        if (lane == 0) asm volatile("global_atomic_add %0, %1, %2, off sc0 sc1" : "=v"(t) : "v"(ctr), "v"(1u) : "memory");
    };
    const char* const qrow = me + OFF_Q + (size_t)src * NMSG * 8;
    const unsigned ibase = (call - 1) * (unsigned)(NPIECE - P0 - NPI) - (P0 + 2 * NPI);
    auto fetch_msg = [&](unsigned info, unsigned p) {
        const unsigned Q = msg_quads(info);
        const char* s = peer + OFF_O + (size_t)(info >> 22) * SLOT + lane * 16;
        for (unsigned i = 0; i < Q; i += 64)
            if (i + lane < Q) glds1(s + i * 16, mlds + p * SLOT + i * 16);
    };
    unsigned cur = 0, par = 0;
    bool early = false;
    char* fp = me + 256;
    if (pj >= NPI) {
        while (ld_sys32(me + OFF_JT + src * 128) - ibase < (unsigned)NPIECE) {
            if (expired(me, call, t0)) { bad = true; break; }
            __builtin_amdgcn_s_sleep(32);
        }
    } else {
        const char* ctr = me + OFF_JT + src * 128;
        u64 ok = 0;
        while (__builtin_amdgcn_s_memrealtime() - t0 < PHOLD) __builtin_amdgcn_s_sleep(2);
        auto fetch = [&](int pi, unsigned p, bool all) {
            int U;
            const int lu = link_unit(pi >> 2, rank, U), st = lu >> 6;
            for (; !((ok >> (U >> 3)) & 1); all = false) {
                u32x4 a = {0u, 0u, 0u, 0u};
                if (all || lane == (U >> 3))
                    asm volatile("global_load_dwordx4 %0, %1, off sc0 sc1\n\ts_waitcnt vmcnt(0)"
                                 : "+&v"(a) : "v"(me + OFF_AF + src * 1024 + lane * 16) : "memory");
                ok |= __builtin_amdgcn_ballot_w64(min(min(a.x, a.y), min(a.z, a.w)) >= call);
                if ((ok >> (U >> 3)) & 1) break;
                if (expired(me, call, t0)) { bad = true; ok = ~0ull; }
                __builtin_amdgcn_s_sleep(AFS);
            }
            const unsigned sidx = (unsigned)((st < 16 ? (rank >> 2) * 16 + st : 16 + (rank & 3) * 8 + st) * 8 + (lu & 7)) * 4 + (pi & 3);
            const char* s = peer + OFF_S + (size_t)sidx * 4096 + lane * 16;
            glds1(s, mlds + p * SLOT); glds1(s + 1024, mlds + p * SLOT + 1024); glds1(s + 2048, mlds + p * SLOT + 2048);
            if (lane < 40) glds1(s + 3072, mlds + p * SLOT + 3072);
            return (((unsigned)(lu + src * 8) * 4 + (pi & 3)) << 16) | sidx;
        };
        unsigned nxt = P0 + NPI + pj;
        u64 e0 = 0;
        cur = fetch(P0 + pj, 0, false);
        {
            u32x4 a;
            asm volatile("global_load_dwordx4 %0, %1, off sc0 sc1\n\ts_waitcnt vmcnt(0)"
                         : "=&v"(a) : "v"(me + OFF_AF + src * 1024 + lane * 16) : "memory");
            ok = __builtin_amdgcn_ballot_w64(min(min(a.x, a.y), min(a.z, a.w)) >= call);
        }
        while (true) {
            char* const m = ml + par * SLOT;
            const unsigned id = cur;
            const bool more = nxt < (unsigned)NPIECE;
            unsigned t = 0;
            par ^= 1;
            if (more) {
                cur = fetch(nxt, par, true); draw(t, ctr);
                asm volatile("global_load_dwordx2 %0, %1, off sc0 sc1" : "=v"(e0) : "v"(qrow + pj * 8) : "memory");
            } else if (__builtin_amdgcn_readfirstlane((unsigned)(e0 >> 32)) >= call) {
                early = true;
                cur = __builtin_amdgcn_readfirstlane((unsigned)e0);
                fetch_msg(cur, par);
            }
            const unsigned esc = *(const unsigned short*)(m + 3584 + 2 * lane);
            *(u64*)(m + 4096 + lane * 8) = 0;
            u32x4 v[4];
#pragma unroll
            for (int h = 0; h < 3; ++h) v[h] = *(const u32x4*)(m + h * 1024 + lane * 16);
            v[3].xy = *(const u32x2*)(m + 3072 + lane * 8);
            const unsigned hdr = __builtin_amdgcn_readfirstlane(esc);
            if (hdr & 64u) {
                WAIT_L(0);
                glds4(peer + OFF_R + (size_t)(id & 2047u) * 4096 + lane * 16, mlds + (par ^ 1) * SLOT);
                WAIT_V(0);
#pragma unroll
                for (int h = 0; h < 4; ++h) v[h] = *(const u32x4*)(m + h * 1024 + lane * 16);
            } else {
                if (lane - 1u < (hdr & 63u))
                    __hip_atomic_fetch_or((unsigned*)(m + 4096 + ((esc >> 6) & 127u) * 4), (esc & 3u) << ((esc >> 1) & 30u), __ATOMIC_RELAXED, __HIP_MEMORY_SCOPE_WAVEFRONT);
                asm volatile("" ::: "memory");
                const u32x2 T = *(const u32x2*)(m + 4096 + lane * 8);
#pragma unroll
                for (int k = 0; k < 2; ++k) {
                    const unsigned g = v[(7 * k + 6) >> 2][(7 * k + 6) & 3];
                    unsigned r = ((g >> 1) & 0x10001000u) | ((g << 1) & 0x80008000u);
#pragma unroll
                    for (int p = 0; p < 6; ++p) r |= (v[(7 * k + p) >> 2][(7 * k + p) & 3] >> (13 - 2 * p)) & (0x00030003u << (2 * p));
                    v[3][2 + k] = r;
                }
                const u16x2 add = __builtin_bit_cast(u16x2, (hdr >> 8) * 0x04000400u);
#pragma unroll
                for (int j = 0; j < 16; ++j) {
                    const unsigned up = (j & 7) < 7 ? T[j >> 3] << (13 - 2 * (j & 7)) : T[j >> 3] >> 1;
                    v[j >> 2][j & 3] = __builtin_bit_cast(unsigned,
                        __builtin_bit_cast(u16x2, (v[j >> 2][j & 3] & 0x9fff9fffu) | (up & 0x60006000u)) + add);
                }
            }
            if (more) WAIT_V(6);
            else if (early && msg_quads(cur) - 193u < 64u) WAIT_V(4);
            else WAIT_V(0);
            st_sys32(fp, call);
            char* d = me + OFF_X + (size_t)(id >> 16) * 4096 + lane * 16;
#pragma unroll
            for (int h = 0; h < 4; ++h) st_wt(d + h * 1024, v[h]);
            fp = me + OFF_LF + (id >> 16) * 4;
            if (!more) break;
            asm volatile("s_waitcnt vmcnt(5)" : "+v"(t), "+v"(e0) :: "memory");
            nxt = __builtin_amdgcn_readfirstlane(t) - ibase;
        }
    }
    {
        const char* ctr = me + OFF_JT + 1024 + src * 128;
        const unsigned base = (call - 1) * (unsigned)(NMSG - 2 * NP) - 3 * NP;
        auto poll = [&](unsigned j) {
            while (true) {
                const u64 e = ld_sys64(qrow + j * 8);
                if (__builtin_amdgcn_readfirstlane((unsigned)(e >> 32)) >= call) return (unsigned)__builtin_amdgcn_readfirstlane((unsigned)e);
                if (bad || expired(me, call, t0)) { bad = true; return 0u; }
                __builtin_amdgcn_s_sleep(4);
            }
        };
        unsigned jB = NP + pj, jC = 2 * NP + pj, eBi = 0;
        bool eBok = false;
        if (!early) {
            WAIT_V(0);
            st_sys32(fp, call);
            cur = poll(pj);
            fetch_msg(cur, par);
        }
        {
            const u64 e = ld_sys64(qrow + jB * 8);
            eBok = __builtin_amdgcn_readfirstlane((unsigned)(e >> 32)) >= call; eBi = __builtin_amdgcn_readfirstlane((unsigned)e);
        }
        WAIT_V(0);
        if (early) st_sys32(fp, call);
        while (true) {
            const unsigned info = cur;
            char* const m = ml + par * SLOT;
            par ^= 1;
            const bool vB = jB < (unsigned)NMSG, vC = jC < (unsigned)NMSG;
            if (vB && eBok) fetch_msg(eBi, par);
            u64 eC = 0; unsigned t = 0;
            if (vC) {
                asm volatile("global_load_dwordx2 %0, %1, off sc0 sc1" : "=v"(eC) : "v"(qrow + jC * 8) : "memory");
                draw(t, ctr);
            }
            const int msg = info >> 22;
            int ti, tj; tile_of(msg >> 3, ti, tj);
            float* o = P.out + (size_t)(2048 * (src >> 2) + 128 * ti + 32 * ((msg & 7) >> 1) + 4 * (lane >> 4)) * GN
                       + 1024 * (src & 3) + 128 * tj + 64 * (msg & 1) + 4 * (lane & 15);
            const unsigned L = min(info & 4095u, 2048u) >> 5, ne = (info >> 12) & 127u;
            const unsigned mask = *(const unsigned*)(m + lane * 4);
            if (ne < 64u) {
                const unsigned esc = *(const unsigned short*)(m + 256 + 104 * L + 4 * (info & 31u) + 2 * lane);
                const unsigned rem = *(const unsigned*)(m + 256 + 104 * L + 4 * (lane & 31));
                const unsigned add = (unsigned)__builtin_amdgcn_readfirstlane(esc) << 23;
                asm volatile("" ::: "memory");
                *(unsigned*)(m + 256 + 128 * L + 4 * (lane & 31)) = (rem & 0x03ffffffu) + add;
                if (lane < L) {
                    char* pl = m + 256 + lane * 8;
                    u32x2 w[13], h[3];
#pragma unroll
                    for (int p = 0; p < 13; ++p) w[p] = *(const u32x2*)(pl + p * 8 * L);
#pragma unroll
                    for (int k = 0; k < 3; ++k) {
                        h[k] = (w[12] >> (2 + 2 * k)) & 0x03000000u;
#pragma unroll
                        for (int c = 0; c < 4; ++c) h[k] |= (w[4 * k + c] >> 26) << (6 * c);
                        *(u32x2*)(pl + (13 + k) * 8 * L) = h[k] + add;
                    }
#pragma unroll
                    for (int p = 0; p < 13; ++p) *(u32x2*)(pl + p * 8 * L) = (w[p] & 0x03ffffffu) + add;
                }
                if (lane - 1u < ne)
                    __hip_atomic_fetch_add((unsigned*)(m + 256 + 4 * (esc & 2047u)), (32u | (esc >> 11)) << 26, __ATOMIC_RELAXED, __HIP_MEMORY_SCOPE_WAVEFRONT);
                asm volatile("" ::: "memory");
            }
            unsigned pre, tot; wave_scan6(__builtin_popcount(mask), pre, tot);
            unsigned pos = 256 + pre * 4;
            u32x4 v[8];
#pragma unroll
            for (int e = 0; e < 32; ++e) {
                v[e >> 2][e & 3] = *(const unsigned*)(m + pos);
                pos += ((mask >> e) & 1u) * 4u;
            }
            asm volatile("" ::: "memory");
#pragma unroll
            for (int e4 = 0; e4 < 8; ++e4) {
#pragma unroll
                for (int c = 0; c < 4; ++c)
                    v[e4][c] &= (unsigned)(((int)(mask << (31 - (4 * e4 + c)))) >> 31);
                st_wt(o + (size_t)((e4 >> 2) * 16 + (e4 & 3)) * GN, v[e4]);
            }
            asm volatile("s_waitcnt vmcnt(8)" : "+v"(eC), "+v"(t) :: "memory");
            if (!vB) break;
            if (!eBok) { eBi = poll(jB); fetch_msg(eBi, par); WAIT_V(0); }
            cur = eBi;
            jB = jC;
            eBok = __builtin_amdgcn_readfirstlane((unsigned)(eC >> 32)) >= call; eBi = __builtin_amdgcn_readfirstlane((unsigned)eC);
            jC = vC ? __builtin_amdgcn_readfirstlane(t) - base : ~0u;
        }
        if (bad) { WAIT_V(0); P.out[(size_t)(2048 * (src >> 2) + 2047) * GN + 1024 * (src & 3) + lane] = NAN; }
    }
}

extern "C" void kernel_launch(int rank, int nranks, ncclComm_t comm,
                              hipStream_t stream,
                              void* const* d_in, const int* in_sizes, int n_in,
                              void* d_out, int out_size,
                              const int* in_dtypes, int out_dtype,
                              void* const* d_out_all,
                              void* const* d_ws_all, size_t ws_bytes) {
    if (nranks != NR || (unsigned)rank >= (unsigned)NR || ws_bytes < WS_NEED || n_in < 2 ||
        in_sizes[0] != 4096 * KL || in_sizes[1] != KL * GN || out_size != 4096 * GN) {
        fprintf(stderr, "unexpected problem shape\n");
        return;
    }
    HIP_CHECK(hipFuncSetAttribute((const void*)gemm_ar_fused, hipFuncAttributeMaxDynamicSharedMemorySize, LDS_BYTES));
    Params P;
    P.x = (const float*)d_in[0];
    P.w = (const float*)d_in[1];
    P.out = (float*)d_out;
    for (int p = 0; p < NR; ++p) P.ws[p] = (char*)d_ws_all[p];
    P.rank = rank;
    gemm_ar_fused<<<NWG, NTHR, LDS_BYTES, stream>>>(P);
    HIP_CHECK(hipGetLastError());
}
```

```cpp
#include <hip/hip_runtime.h>
#include <rccl/rccl.h>
#include <cstdio>
#include <cstdint>

typedef unsigned u32x4 __attribute__((ext_vector_type(4)));
typedef unsigned u32x2 __attribute__((ext_vector_type(2)));
typedef unsigned short u16x2 __attribute__((ext_vector_type(2)));
typedef float f32x4 __attribute__((ext_vector_type(4)));
typedef float f32x2 __attribute__((ext_vector_type(2)));
typedef _Float16 f16x8 __attribute__((ext_vector_type(8)));
typedef unsigned long long u64;

#define HIP_CHECK(expr) do { hipError_t _e = (expr); if (_e != hipSuccess) { \
    fprintf(stderr, "HIP error %s at %s:%d\n", hipGetErrorString(_e), __FILE__, __LINE__); \
    return; } } while (0)

constexpr int NR = 8;
constexpr int GN = 4096;
constexpr int KL = 512;
constexpr int NWG = 256, NTHR = 512;
#define NP_ 28
#define NPI_ 26
#define DLY0 64
#define DLYA 150
#define DLYS 2
#define DLYM 600
#define PSL 16
#define PHOLD 400
#define AFG 48
#define AFS 1
#define WPB 2
constexpr int NP = NP_;
constexpr int NPI = NPI_;
constexpr int NSEND = 112;
constexpr int NGEMM = 128;
constexpr int NMSG = NGEMM * 8;
constexpr int NPIECE = 768;
constexpr size_t UNIT = 16384;
constexpr size_t SLOT = 9216;
constexpr unsigned long long DEADLINE = 40000000ull;
#define PUSHB 1
#define NE 8
constexpr int NPUSH = (16 + NE) * PUSHB;
constexpr int P0 = 4 * (8 + NE) * PUSHB;


static_assert(NPUSH == 24 && WPB == 2 && NP <= 32 && NPI > 16 && NPI <= NP && AFG % 16 == 0 && AFG >= NPUSH, "layout");

constexpr size_t OFF_CNT   = 57344;
constexpr size_t OFF_TMO   = 4096;
constexpr size_t OFF_TICK  = 6144;
constexpr size_t OFF_LF    = 16384;
constexpr size_t OFF_AF    = 49152;
constexpr size_t OFF_AC    = 12288;
constexpr size_t OFF_JT    = 8192;
constexpr size_t OFF_Q     = 65536;
constexpr size_t OFF_X     = 1u << 20;
constexpr size_t OFF_S     = OFF_X + 24 * 64 * UNIT;
constexpr size_t OFF_O     = OFF_S + 512 * UNIT;
constexpr size_t OFF_R     = OFF_O + (size_t)NMSG * SLOT;
constexpr size_t WS_NEED   = OFF_R + 512 * UNIT;

constexpr int LDS_STAGE = 32768, LDS_MISC = 4 * LDS_STAGE, LDS_SNAP = LDS_MISC + 1024, LDS_BYTES = LDS_SNAP + 2048;
constexpr int WAVE_LDS = 16384;

struct Params { const float* x; const float* w; float* out; char* ws[NR]; int rank; };

__device__ __forceinline__ char* ws_sel(const Params& P, int d) {
    char* r = P.ws[0];
#pragma unroll
    for (int p = 1; p < NR; ++p) { char* w = P.ws[p]; asm("" : "+s"(w)); r = d == p ? w : r; }
    return r;
}
__device__ __forceinline__ void st_wt(void* p, u32x4 v) {
    asm volatile("global_store_dwordx4 %0, %1, off sc0 sc1\n\ts_nop 1" :: "v"(p), "v"(v) : "memory");
}
__device__ __forceinline__ void glds1(const void* g, unsigned lds) {
    unsigned keep;
    asm volatile("s_mov_b32 %0, m0\n\ts_mov_b32 m0, %2\n\ts_nop 0\n\t"
                 "global_load_lds_dwordx4 %1, off sc0 sc1\n\ts_mov_b32 m0, %0"
                 : "=&s"(keep) : "v"(g), "s"(lds) : "memory");
}
__device__ __forceinline__ void glds4(const void* g, unsigned lds) {
    unsigned keep;
    asm volatile("s_mov_b32 %0, m0\n\ts_mov_b32 m0, %2\n\ts_nop 0\n\t"
                 "global_load_lds_dwordx4 %1, off sc0 sc1\n\t"
                 "global_load_lds_dwordx4 %1, off offset:1024 sc0 sc1\n\t"
                 "global_load_lds_dwordx4 %1, off offset:2048 sc0 sc1\n\t"
                 "global_load_lds_dwordx4 %1, off offset:3072 sc0 sc1\n\t"
                 "s_mov_b32 m0, %0"
                 : "=&s"(keep) : "v"(g), "s"(lds) : "memory");
}
#define WAIT_V(n) asm volatile("s_waitcnt vmcnt(" #n ")" ::: "memory")
#define WAIT_L(n) asm volatile("s_waitcnt lgkmcnt(" #n ")" ::: "memory")
#define BAR() do { asm volatile("s_waitcnt lgkmcnt(0)" ::: "memory"); __builtin_amdgcn_s_barrier(); asm volatile("" ::: "memory"); } while (0)

__device__ __forceinline__ u64 ld_sys64(const void* p) {
    return __hip_atomic_load((const u64*)p, __ATOMIC_RELAXED, __HIP_MEMORY_SCOPE_SYSTEM);
}
__device__ __forceinline__ void st_sys64(void* p, u64 v) {
    asm volatile("global_store_dwordx2 %0, %1, off sc0 sc1\n\ts_nop 1" :: "v"(p), "v"(v) : "memory");
}
__device__ __forceinline__ unsigned ld_sys32(const void* p) {
    return __hip_atomic_load((const unsigned*)p, __ATOMIC_RELAXED, __HIP_MEMORY_SCOPE_SYSTEM);
}
__device__ __forceinline__ void st_sys32(void* p, unsigned v) {
    asm volatile("global_store_dword %0, %1, off sc0 sc1\n\ts_nop 1" :: "v"(p), "v"(v) : "memory");
}
__device__ __forceinline__ unsigned ld_ag32(const void* p) {
    return __hip_atomic_load((const unsigned*)p, __ATOMIC_RELAXED, __HIP_MEMORY_SCOPE_AGENT);
}
__device__ __forceinline__ void st_ag32(void* p, unsigned v) {
    __hip_atomic_store((unsigned*)p, v, __ATOMIC_RELAXED, __HIP_MEMORY_SCOPE_AGENT);
}
__device__ __forceinline__ bool expired(char* me, unsigned call, u64 t0) {
    const u64 dt = __builtin_amdgcn_s_memrealtime() - t0;
    if (dt < 100000) return false;
    bool e = ld_ag32(me + OFF_TMO) == call || dt > DEADLINE;
    e = __builtin_amdgcn_readfirstlane(e);
    if (e) st_ag32(me + OFF_TMO, call);
    return e;
}
__device__ __forceinline__ void wave_scan6(unsigned c, unsigned& pre, unsigned& tot) {
    pre = 0; tot = 0;
#pragma unroll
    for (int b = 0; b < 6; ++b) {
        u64 m = __builtin_amdgcn_ballot_w64((c >> b) & 1);
        unsigned p = __builtin_amdgcn_mbcnt_hi((unsigned)(m >> 32), __builtin_amdgcn_mbcnt_lo((unsigned)m, 0));
        pre += p << b;
        tot += (unsigned)__builtin_popcountll(m) << b;
    }
}
__device__ __forceinline__ void tile_of(int g, int& i, int& j) {
    int xcd = g & 7, t = g >> 3;
    i = 2 * (t >> 1) + (xcd >> 2);
    j = 2 * (xcd & 3) + (t & 1);
}

#define LAG 8
template <int NA, int NL> __device__ __forceinline__ void seq_of(int U, bool& late, int& ks, int& o) {
    if (U < NA * LAG) { ks = U / NA; o = U - NA * ks; late = false; }
    else if (U < NA * LAG + (NA + NL) * (8 - LAG)) {
        const int V = U - NA * LAG, s = V / (NA + NL); o = V - s * (NA + NL);
        late = o < NL;
        if (late) ks = s; else { ks = s + LAG; o -= NL; }
    } else {
        const int V = U - (NA * LAG + (NA + NL) * (8 - LAG)), s = V / NL;
        o = V - NL * s; ks = 8 - LAG + s; late = true;
    }
}
__device__ __forceinline__ void unit_of(int U, bool& isx, int& grp, int& strip, int& ks) {
    bool late; int o; seq_of<32 + 2 * NE, 32 - 2 * NE>(U, late, ks, o);
    isx = late || o < 2 * NE;
    if (isx) { grp = o & 1; strip = (late ? NE : 0) + (o >> 1); }
    else     { o -= 2 * NE; grp = ((o >> 3) + o) & 3; strip = o & 7; }
}
__device__ __forceinline__ unsigned msg_quads(unsigned info) {
    const unsigned n = min(info & 4095u, 2048u), ne = (info >> 12) & 127u;
    return 16 + (ne > 63u ? (n + 3) >> 2 : (104 * (n >> 5) + 4 * (n & 31u) + 2 * ne + 17) >> 4);
}
__device__ __forceinline__ int link_unit(int L, int rank, int& U) {
    bool late; int ks, o; seq_of<8 + NE, 16 - NE>(L, late, ks, o);
    const int na = late ? min(ks + LAG, 8) : ks, nl = late ? ks : max(ks + 1 - LAG, 0);
    U = na * (32 + 2 * NE) + nl * (32 - 2 * NE)
        + (late || o < NE ? 2 * o + (rank >> 2) : NE + o + 8 * ((rank - o + NE) & 3));
    return (late ? NE + o : o < NE ? o : 16 - NE + o) * 64 + ks;
}

__global__ __launch_bounds__(NTHR, 2) void gemm_ar_fused(Params P) {
    extern __shared__ __attribute__((aligned(16))) char smem[];
    const int tid = threadIdx.x, lane = tid & 63;
    const int wv = __builtin_amdgcn_readfirstlane(tid >> 6);
    const int bid = blockIdx.x, b = (bid - (128 - NSEND)) & 255, rank = P.rank;
    char* const me = P.ws[rank];
    const unsigned lds0 = (unsigned)(uintptr_t)(__attribute__((address_space(3))) char*)smem;
    const u64 t0 = __builtin_amdgcn_s_memrealtime();

    unsigned* misc = (unsigned*)(smem + LDS_MISC);
    unsigned cnt = ld_ag32(me + OFF_CNT + 4 * (bid * 8 + wv));
    if (tid < 4) misc[tid] = 0;
    if (tid < 128) *(u32x4*)(smem + LDS_SNAP + tid * 16) = (u32x4){0u, 0u, 0u, 0u};
    BAR();
    const bool pblk = b < NSEND;
    const int cb = pblk ? bid + 128 : bid < 128 ? bid : bid >= 248 ? bid - 232 : bid - 104;
    if (cb >= DLY0 && !(pblk && wv < WPB))
        while (__builtin_amdgcn_s_memrealtime() - t0 < (unsigned)min(DLYA + DLYS * (cb - DLY0), DLYM)) __builtin_amdgcn_s_sleep(4);
    for (int p = pblk ? wv - WPB : wv; p >= 0 && p < 8; p += pblk ? 8 - WPB : 8) {
        const int pc = cb < NPUSH ? p * NPUSH + cb : cb * 8 + p;
        const int U = pc >> 2, qd = pc & 3;
        bool isx; int grp, strip, ks; unit_of(U, isx, grp, strip, ks);
        f16x8 c[4];
        if (isx) {
            const int kg = (lane & 7) ^ (lane >> 3);
            const float* s = P.x + (size_t)(2048 * grp + 128 * strip + 32 * qd + (lane >> 3)) * KL + 64 * ks + 4 * kg;
            f32x4 f[8];
#pragma unroll
            for (int t = 0; t < 8; ++t) f[t] = *(const f32x4*)(s + (t >> 1) * 8 * KL + (t & 1) * 32);
#pragma unroll
            for (int e = 0; e < 32; ++e) c[e >> 3][e & 7] = (_Float16)f[e >> 2][e & 3];
        } else {
            const float* s = P.w + (size_t)(64 * ks + 8 * qd) * GN + 1024 * grp + 128 * strip + 4 * (lane & 31) + 2 * (lane >> 5);
            f32x2 f[16];
#pragma unroll
            for (int e = 0; e < 16; ++e) f[e] = *(const f32x2*)(s + (size_t)((e & 3) + ((e >> 2) & 1) * 32 + (e >> 3) * 4) * GN);
#pragma unroll
            for (int e = 0; e < 32; ++e) c[e >> 3][e & 7] = (_Float16)f[(e >> 4) * 8 + (e & 7)][(e >> 3) & 1];
        }
        asm volatile("" : "+v"(cnt));
        const unsigned call = cnt + 1;
        const int st = isx ? strip : 16 + strip;
        const size_t li = (size_t)(st * 64 + rank * 8 + ks) * 4 + qd;
        const bool push = cb < NPUSH;
        const bool mine = grp == (isx ? rank >> 2 : rank & 3);
        if (push) {
#pragma unroll
            for (int h = 0; h < 4; ++h) {
                for (int t = 0; t < (isx ? 4 : 2); ++t) {
                    const unsigned d = (unsigned)(isx ? 4 * grp + ((t + qd) & 3) : grp + 4 * ((t + qd) & 1)) & (NR - 1);
                    st_wt(P.ws[d] + OFF_X + li * 4096 + lane * 16 + h * 1024, __builtin_bit_cast(u32x4, c[h]));
                }
            }
        } else {
            const size_t so = (size_t)(((isx ? grp * 16 + strip : 32 + grp * 8 + strip) * 8 + ks) * 4 + qd) * 4096;
            unsigned d[16], T[2] = {0u, 0u};
            u16x2 mx = {0, 0};
#pragma unroll
            for (int j = 0; j < 16; ++j) {
                d[j] = __builtin_bit_cast(u32x4, c[j >> 2])[j & 3];
                mx = __builtin_elementwise_max(mx, __builtin_bit_cast(u16x2, d[j] & 0x7fff7fffu));
            }
            unsigned em = max(mx.x, mx.y);
#pragma unroll
            for (int o = 32; o > 0; o >>= 1) em = max(em, (unsigned)__shfl_xor((int)em, o));
            const unsigned low = max((unsigned)__builtin_amdgcn_readfirstlane(em) >> 10, 7u) - 7u;
#pragma unroll
            for (int j = 0; j < 16; ++j) {
                d[j] = __builtin_bit_cast(unsigned, __builtin_bit_cast(u16x2, d[j]) - __builtin_bit_cast(u16x2, low * 0x04000400u));
                T[j >> 3] |= ((d[j] >> 13) & 0x00030003u) << (2 * (j & 7));
            }
            u64 z = ((u64)T[1] << 32) | T[0];
            u64 nz = (z | (z >> 1)) & 0x5555555555555555ull;
            unsigned epre, nesc; wave_scan6(__builtin_popcountll(nz), epre, nesc);
            char* el = smem + LDS_MISC - 2048 + wv * 256;
            if (nesc > 63u) {
#pragma unroll
                for (int h = 0; h < 4; ++h) st_wt(me + OFF_R + so + h * 1024 + lane * 16, __builtin_bit_cast(u32x4, c[h]));
            } else {
                for (; nz; nz &= nz - 1) {
                    const int bit = __builtin_ctzll(nz);
                    *(unsigned short*)(el + 2 * ++epre) = (lane << 7) | (bit << 1) | ((unsigned)(z >> bit) & 3u);
                }
#pragma unroll
                for (int j = 0; j < 14; ++j) {
                    const unsigned g = d[14 + j / 7], p = j % 7;
                    const unsigned top = p < 6 ? g << (13 - 2 * p) : ((g << 1) & 0x20002000u) | ((g >> 1) & 0x40004000u);
                    d[j] = (d[j] & 0x9fff9fffu) | (top & 0x60006000u);
                }
#pragma unroll
                for (int h = 0; h < 3; ++h)
                    st_wt(me + OFF_S + so + h * 1024 + lane * 16, (u32x4){d[4 * h], d[4 * h + 1], d[4 * h + 2], d[4 * h + 3]});
                st_sys64(me + OFF_S + so + 3072 + lane * 8, ((u64)d[13] << 32) | d[12]);
            }
            if (lane == 0) *(unsigned short*)el = min(nesc, 64u) | (low << 8);
            asm volatile("" ::: "memory");
            if (lane < 8) st_wt(me + OFF_S + so + 3584 + lane * 16, *(const u32x4*)(el + lane * 16));
            if (mine) {
#pragma unroll
                for (int h = 0; h < 4; ++h) st_wt(me + OFF_X + li * 4096 + lane * 16 + h * 1024, __builtin_bit_cast(u32x4, c[h]));
            }
        }
        WAIT_V(0);
        if (push) {
            if (lane < (isx ? 4 : 2)) st_sys32(ws_sel(P, isx ? 4 * grp + lane : grp + 4 * lane) + OFF_LF + li * 4, call);
        } else {
            unsigned k = 0;
            if (lane == 0) k = __hip_atomic_fetch_add(&misc[1], 1u, __ATOMIC_RELAXED, __HIP_MEMORY_SCOPE_WORKGROUP);
            if (__builtin_amdgcn_readfirstlane(k) == 7) {
                if (cb < AFG) {
                    if (lane < NR && lane != rank) st_sys32(ws_sel(P, lane) + OFF_AF + (rank * 256 + cb) * 4, call);
                } else {
                    if (lane == 0) k = __hip_atomic_fetch_add((unsigned*)(me + OFF_AC + (cb >> 4) * 128), 1u, __ATOMIC_RELAXED, __HIP_MEMORY_SCOPE_AGENT);
                    if ((__builtin_amdgcn_readfirstlane(k) & 15) == 15 && lane < 4 * NR && (lane >> 2) != rank)
                        st_wt(ws_sel(P, lane >> 2) + OFF_AF + (rank * 256 + (cb & ~15)) * 4 + (lane & 3) * 16, (u32x4){call, call, call, call});
                }
            }
            if (mine && lane == 0) st_sys32(me + OFF_LF + li * 4, call);
        }
    }
    const unsigned call = __builtin_amdgcn_readfirstlane(cnt) + 1;
    if (lane == 0) st_ag32(me + OFF_CNT + 4 * (bid * 8 + wv), call);

    if (bid >= 128) {
        const int g = bid & 127;
        int ti, tj; tile_of(g, ti, tj);
        const size_t fo = (size_t)((rank + lane) & 7) * 128 + (lane >> 3) * 16;
        const char* fw = me + OFF_LF + (size_t)(16 + tj) * 1024;
        const char* fx = me + OFF_LF + (size_t)ti * 1024;
        u64 avail = 0, left = ~0ull;
        constexpr int nst = 64;
        auto arrived = [&](u32x4 a, u32x4 c) {
            return left & __builtin_amdgcn_ballot_w64(min(min(min(a.x, a.y), min(a.z, a.w)), min(min(c.x, c.y), min(c.z, c.w))) >= call);
        };
        auto pick = [&](int kk) {
            if (!avail) avail = arrived(*(const u32x4*)(smem + LDS_SNAP + fo), *(const u32x4*)(smem + LDS_SNAP + 1024 + fo));
            while (!avail) {
                u32x4 a, c;
                asm volatile("global_load_dwordx4 %0, %2, off sc0 sc1\n\tglobal_load_dwordx4 %1, %3, off sc0 sc1\n\ts_waitcnt vmcnt(0)"
                             : "=&v"(a), "=&v"(c) : "v"(fw + fo), "v"(fx + fo) : "memory");
                avail = arrived(a, c);
                if (avail) break;
                if (expired(me, call, t0)) { avail = left; misc[2] = 1; break; }
                __builtin_amdgcn_s_sleep(PSL);
            }
            const int id = __builtin_ctzll(avail);
            avail &= avail - 1; left &= ~(1ull << id);
            if (lane == 0) misc[8 + (kk & 7)] = ((rank + id) & 7) * 8 + (id >> 3);
        };
        const int kh = wv >> 2, wr = (wv & 2) + kh, wc = wv & 1, q = lane >> 4, r = lane & 15;
        f32x4 acc[2][4];
        {
            f32x4 ac[4][4];
#pragma unroll
            for (int mi = 0; mi < 4; ++mi)
#pragma unroll
                for (int nn = 0; nn < 4; ++nn) ac[mi][nn] = (f32x4){0.f, 0.f, 0.f, 0.f};
            const char* gsrc = me + OFF_X + (size_t)(wv < 4 ? ti : 16 + tj) * 64 * UNIT
                               + (wv & 3) * 4096 + lane * 16;
            const unsigned ldst = lds0 + wv * 4096;
            auto stage = [&](int kk) {
                if (wv == 0) glds1(((kk & 1) ? fw : fx) + lane * 16, lds0 + LDS_SNAP + (kk & 1) * 1024);
                glds4(gsrc + (size_t)(misc[8 + (kk & 7)] & 63u) * UNIT, ldst + (kk & 3) * LDS_STAGE);
            };
            for (int kk = 0; kk < 3; ++kk) {
                if (wv == 0) pick(kk);
                BAR();
                stage(kk);
            }
            const int aoff = (32 * (wv & 2) + r) * 128 + ((4 * kh + q) ^ (r & 7)) * 16;
            const int boff = 16384 + kh * 8192 + q * 2048 + (16 * wc + r) * 16;
            for (int kk = 0; kk < nst; ++kk) {
                if (wv == 0 && kk + 3 < nst) pick(kk + 3);
                if (kk + 2 < nst) { if (wv == 0) WAIT_V(10); else WAIT_V(8); }
                else if (kk + 1 < nst) WAIT_V(4); else WAIT_V(0);
                BAR();
                if (kk + 3 < nst) stage(kk + 3);
                const char* st = smem + (kk & 3) * LDS_STAGE;
                f16x8 af[4], bf[4];
#pragma unroll
                for (int i = 0; i < 4; ++i) {
                    af[i] = *(const f16x8*)(st + aoff + i * 2048);
                    bf[i] = *(const f16x8*)(st + boff + (i & 1) * 1024 + (i >> 1) * 512);
                }
#pragma unroll
                for (int mi = 0; mi < 4; ++mi)
#pragma unroll
                    for (int nn = 0; nn < 4; ++nn)
                        ac[mi][nn] = __builtin_amdgcn_mfma_f32_16x16x32_f16(af[mi], bf[nn], ac[mi][nn], 0, 0, 0);
            }
            BAR();
#pragma unroll
            for (int i = 0; i < 8; ++i)
                *(f32x4*)(smem + (wv ^ 4) * WAVE_LDS + i * 1024 + lane * 16) = kh ? ac[i >> 2][i & 3] : ac[2 + (i >> 2)][i & 3];
            BAR();
#pragma unroll
            for (int i = 0; i < 8; ++i)
                acc[i >> 2][i & 3] = (kh ? ac[2 + (i >> 2)][i & 3] : ac[i >> 2][i & 3])
                                     + *(const f32x4*)(smem + wv * WAVE_LDS + i * 1024 + lane * 16);
        }
        const bool poison = misc[2];
        unsigned qi = 0;
        if (tid == 0) asm volatile("global_atomic_add %0, %1, %2, off sc0 sc1" : "=v"(qi) : "v"(me + OFF_TICK), "v"(1u) : "memory");
        if (poison) {
#pragma unroll
            for (int i = 0; i < 8; ++i) acc[i >> 2][i & 3] = (f32x4){NAN, NAN, NAN, NAN};
        }
        unsigned u[32], mask = 0, umax = 0;
#pragma unroll
        for (int mi = 0; mi < 2; ++mi)
#pragma unroll
            for (int j = 0; j < 4; ++j)
#pragma unroll
                for (int nn = 0; nn < 4; ++nn) {
                    const int e = (mi * 4 + j) * 4 + nn;
                    const float v = acc[mi][nn][j];
                    const bool keep = !(v <= 0.f);
                    u[e] = keep ? __float_as_uint(v) : 0u;
                    mask |= (keep ? 1u : 0u) << e;
                    umax = max(umax, u[e]);
                }
#pragma unroll
        for (int o = 32; o > 0; o >>= 1) umax = max(umax, (unsigned)__shfl_xor((int)umax, o));
        umax = __builtin_amdgcn_readfirstlane(umax);
        const unsigned ebase = max(umax >> 23, 7u) - 7u;
        unsigned em = 0;
#pragma unroll
        for (int e = 0; e < 32; ++e) em |= (u[e] < (ebase << 23) ? 1u : 0u) << e;
        em &= mask;
        unsigned pre, n, epre, nesc;
        wave_scan6(__builtin_popcount(mask), pre, n);
        wave_scan6(__builtin_popcount(em), epre, nesc);
        const bool raw = nesc > 63u || (umax >> 31);
        const unsigned sub = raw ? 0u : ebase << 23;
        char* ml = smem + wv * WAVE_LDS;
        *(unsigned*)(ml + lane * 4) = mask;
        {
            unsigned pos = 256 + pre * 4;
#pragma unroll
            for (int e = 0; e < 32; ++e) {
                const bool k = (mask >> e) & 1;
                *(unsigned*)(ml + (k ? pos : 12288u + lane * 4)) = u[e] - sub;
                pos += k ? 4 : 0;
            }
        }
        if (!raw) {
            const unsigned L = n >> 5;
            if (lane == 0) *(unsigned short*)(ml + 13312) = ebase;
            for (; em; em &= em - 1) {
                const unsigned pos = pre + __builtin_popcount(mask & ((em & -em) - 1u));
                *(unsigned short*)(ml + 13312 + 2 * ++epre) = pos | ((*(const unsigned*)(ml + 256 + 4 * pos) >> 26) << 11);
            }
            if (lane < L) {
                char* pl = ml + 256 + lane * 8;
                u32x2 h[3];
#pragma unroll
                for (int k = 0; k < 3; ++k) h[k] = *(const u32x2*)(pl + (13 + k) * 8 * L);
#pragma unroll
                for (int p = 0; p < 13; ++p) {
                    u32x2* a = (u32x2*)(pl + p * 8 * L);
                    const u32x2 top = p < 12 ? h[p >> 2] << (26 - 6 * (p & 3))
                                             : ((h[0] << 2) & 0x0c000000u) | ((h[1] << 4) & 0x30000000u) | ((h[2] << 6) & 0xc0000000u);
                    *a = (*a & 0x03ffffffu) | (top & 0xfc000000u);
                }
            }
            asm volatile("" ::: "memory");
            *(unsigned*)(ml + 256 + 104 * L + 4 * (lane & 31)) = *(const unsigned*)(ml + 256 + 128 * L + 4 * (lane & 31));
            asm volatile("" ::: "memory");
            *(unsigned short*)(ml + 256 + 104 * L + 4 * (n & 31u) + 2 * lane) = *(const unsigned short*)(ml + 13312 + 2 * lane);
        }
        const unsigned info = n | ((raw ? 64u : nesc) << 12) | ((unsigned)(g * 8 + wr * 2 + wc) << 22);
        const unsigned Q = msg_quads(info);
        const int msg = g * 8 + wr * 2 + wc;
        {
            char* dst = me + OFF_O + (size_t)msg * SLOT + lane * 16;
            for (unsigned i = 0; i < Q; i += 64)
                if (i + lane < Q) st_wt(dst + i * 16, *(const u32x4*)(ml + (i + lane) * 16));
        }
        asm volatile("s_waitcnt vmcnt(0)" : "+v"(qi) :: "memory");
        unsigned k = 0;
        if (tid == 0) misc[24] = qi;
        if (lane == 0) {
            misc[16 + wv] = info;
            asm volatile("" ::: "memory");
            k = __hip_atomic_fetch_add(&misc[3], 1u, __ATOMIC_RELAXED, __HIP_MEMORY_SCOPE_WORKGROUP);
        }
        if (__builtin_amdgcn_readfirstlane(k) == 7 && (lane >> 3) != rank) {
            const volatile unsigned* mv = misc;
            st_sys64(ws_sel(P, lane >> 3) + OFF_Q + (size_t)(rank * NMSG + (mv[24] & (NGEMM - 1)) * 8 + (lane & 7)) * 8,
                     ((u64)call << 32) | mv[16 + (lane & 7)]);
        }
        {
            float* o = P.out + (size_t)(2048 * (rank >> 2) + 128 * ti + 32 * wr + 4 * q) * GN
                       + 1024 * (rank & 3) + 128 * tj + 64 * wc + 4 * r;
#pragma unroll
            for (int e4 = 0; e4 < 8; ++e4)
                st_wt(o + (size_t)((e4 >> 2) * 16 + (e4 & 3)) * GN, (u32x4){u[4 * e4], u[4 * e4 + 1], u[4 * e4 + 2], u[4 * e4 + 3]});
        }
        return;
    }
    const int pj = b / 7 + 16 * wv, pw = 7 * pj + (b + wv) % 7;
    if (!pblk) return;
    if (wv >= WPB || pj >= NP) return;

    const unsigned src = (unsigned)(rank + 1 + pw - 7 * pj) & (NR - 1);
    char* const peer = P.ws[src];
    char* const ml = smem + wv * 2 * SLOT;
    const unsigned mlds = lds0 + wv * 2 * SLOT;
    bool bad = false;
    auto draw = [&](unsigned& t, const char* ctr) {
        if (lane == 0) asm volatile("global_atomic_add %0, %1, %2, off sc0 sc1" : "=v"(t) : "v"(ctr), "v"(1u) : "memory");
    };
    const char* const qrow = me + OFF_Q + (size_t)src * NMSG * 8;
    const unsigned ibase = (call - 1) * (unsigned)(NPIECE - P0 - NPI) - (P0 + 2 * NPI);
    auto fetch_msg = [&](unsigned info, unsigned p) {
        const unsigned Q = msg_quads(info);
        const char* s = peer + OFF_O + (size_t)(info >> 22) * SLOT + lane * 16;
        for (unsigned i = 0; i < Q; i += 64)
            if (i + lane < Q) glds1(s + i * 16, mlds + p * SLOT + i * 16);
    };
    unsigned cur = 0, par = 0;
    bool early = false;
    char* fp = me + 256;
    if (pj >= NPI) {
        while (ld_sys32(me + OFF_JT + src * 128) - ibase < (unsigned)NPIECE) {
            if (expired(me, call, t0)) { bad = true; break; }
            __builtin_amdgcn_s_sleep(32);
        }
    } else {
        const char* ctr = me + OFF_JT + src * 128;
        u64 ok = 0;
        while (__builtin_amdgcn_s_memrealtime() - t0 < PHOLD) __builtin_amdgcn_s_sleep(2);
        auto fetch = [&](int pi, unsigned p, bool all) {
            int U;
            const int lu = link_unit(pi >> 2, rank, U), st = lu >> 6;
            for (; !((ok >> (U >> 3)) & 1); all = false) {
                u32x4 a = {0u, 0u, 0u, 0u};
                if (all || lane == (U >> 3))
                    asm volatile("global_load_dwordx4 %0, %1, off sc0 sc1\n\ts_waitcnt vmcnt(0)"
                                 : "+&v"(a) : "v"(me + OFF_AF + src * 1024 + lane * 16) : "memory");
                ok |= __builtin_amdgcn_ballot_w64(min(min(a.x, a.y), min(a.z, a.w)) >= call);
                if ((ok >> (U >> 3)) & 1) break;
                if (expired(me, call, t0)) { bad = true; ok = ~0ull; }
                __builtin_amdgcn_s_sleep(AFS);
            }
            const unsigned sidx = (unsigned)((st < 16 ? (rank >> 2) * 16 + st : 16 + (rank & 3) * 8 + st) * 8 + (lu & 7)) * 4 + (pi & 3);
            const char* s = peer + OFF_S + (size_t)sidx * 4096 + lane * 16;
            glds1(s, mlds + p * SLOT); glds1(s + 1024, mlds + p * SLOT + 1024); glds1(s + 2048, mlds + p * SLOT + 2048);
            if (lane < 40) glds1(s + 3072, mlds + p * SLOT + 3072);
            return (((unsigned)(lu + src * 8) * 4 + (pi & 3)) << 16) | sidx;
        };
        unsigned nxt = P0 + NPI + pj;
        auto peek = [&] { if (lane == 0) glds1(qrow + (pj >> 1) * 16, lds0 + 65536 + wv * 16); };
        auto first = [&] {
            const u32x2 e = *(const volatile u32x2*)(smem + 65536 + wv * 16 + (pj & 1) * 8);
            early = (unsigned)__builtin_amdgcn_readfirstlane(e.y) >= call;
            if (early) { cur = __builtin_amdgcn_readfirstlane(e.x); fetch_msg(cur, par); }
        };
        cur = fetch(P0 + pj, 0, false);
        {
            u32x4 a;
            asm volatile("global_load_dwordx4 %0, %1, off sc0 sc1\n\ts_waitcnt vmcnt(0)"
                         : "=&v"(a) : "v"(me + OFF_AF + src * 1024 + lane * 16) : "memory");
            ok = __builtin_amdgcn_ballot_w64(min(min(a.x, a.y), min(a.z, a.w)) >= call);
        }
        while (true) {
            char* const m = ml + par * SLOT;
            const unsigned id = cur;
            const bool more = nxt < (unsigned)NPIECE;
            unsigned t = 0;
            par ^= 1;
            if (more) { cur = fetch(nxt, par, true); draw(t, ctr); }
            else first();
            if (!early) peek();
            const unsigned esc = *(const unsigned short*)(m + 3584 + 2 * lane);
            *(u64*)(m + 4096 + lane * 8) = 0;
            u32x4 v[4];
#pragma unroll
            for (int h = 0; h < 3; ++h) v[h] = *(const u32x4*)(m + h * 1024 + lane * 16);
            v[3].xy = *(const u32x2*)(m + 3072 + lane * 8);
            const unsigned hdr = __builtin_amdgcn_readfirstlane(esc);
            if (hdr & 64u) {
                WAIT_L(0);
                glds4(peer + OFF_R + (size_t)(id & 2047u) * 4096 + lane * 16, mlds + (par ^ 1) * SLOT);
                WAIT_V(0);
#pragma unroll
                for (int h = 0; h < 4; ++h) v[h] = *(const u32x4*)(m + h * 1024 + lane * 16);
            } else {
                if (lane - 1u < (hdr & 63u))
                    __hip_atomic_fetch_or((unsigned*)(m + 4096 + ((esc >> 6) & 127u) * 4), (esc & 3u) << ((esc >> 1) & 30u), __ATOMIC_RELAXED, __HIP_MEMORY_SCOPE_WAVEFRONT);
                asm volatile("" ::: "memory");
                const u32x2 T = *(const u32x2*)(m + 4096 + lane * 8);
#pragma unroll
                for (int k = 0; k < 2; ++k) {
                    const unsigned g = v[(7 * k + 6) >> 2][(7 * k + 6) & 3];
                    unsigned r = ((g >> 1) & 0x10001000u) | ((g << 1) & 0x80008000u);
#pragma unroll
                    for (int p = 0; p < 6; ++p) r |= (v[(7 * k + p) >> 2][(7 * k + p) & 3] >> (13 - 2 * p)) & (0x00030003u << (2 * p));
                    v[3][2 + k] = r;
                }
                const u16x2 add = __builtin_bit_cast(u16x2, (hdr >> 8) * 0x04000400u);
#pragma unroll
                for (int j = 0; j < 16; ++j) {
                    const unsigned up = (j & 7) < 7 ? T[j >> 3] << (13 - 2 * (j & 7)) : T[j >> 3] >> 1;
                    v[j >> 2][j & 3] = __builtin_bit_cast(unsigned,
                        __builtin_bit_cast(u16x2, (v[j >> 2][j & 3] & 0x9fff9fffu) | (up & 0x60006000u)) + add);
                }
            }
            if (more) WAIT_V(6);
            else if (early && msg_quads(cur) - 193u < 64u) WAIT_V(4);
            else {
                WAIT_V(0);
                if (!early) first();
            }
            st_sys32(fp, call);
            char* d = me + OFF_X + (size_t)(id >> 16) * 4096 + lane * 16;
#pragma unroll
            for (int h = 0; h < 4; ++h) st_wt(d + h * 1024, v[h]);
            fp = me + OFF_LF + (id >> 16) * 4;
            if (!more) break;
            asm volatile("s_waitcnt vmcnt(5)" : "+v"(t) :: "memory");
            nxt = __builtin_amdgcn_readfirstlane(t) - ibase;
        }
    }
    {
        const char* ctr = me + OFF_JT + 1024 + src * 128;
        const unsigned base = (call - 1) * (unsigned)(NMSG - 2 * NP) - 3 * NP;
        auto poll = [&](unsigned j) {
            while (true) {
                const u64 e = ld_sys64(qrow + j * 8);
                if (__builtin_amdgcn_readfirstlane((unsigned)(e >> 32)) >= call) return (unsigned)__builtin_amdgcn_readfirstlane((unsigned)e);
                if (bad || expired(me, call, t0)) { bad = true; return 0u; }
                __builtin_amdgcn_s_sleep(4);
            }
        };
        unsigned jB = NP + pj, jC = 2 * NP + pj, eBi = 0;
        bool eBok = false;
        if (!early) {
            WAIT_V(0);
            st_sys32(fp, call);
            cur = poll(pj);
            fetch_msg(cur, par);
        }
        {
            const u64 e = ld_sys64(qrow + jB * 8);
            eBok = __builtin_amdgcn_readfirstlane((unsigned)(e >> 32)) >= call; eBi = __builtin_amdgcn_readfirstlane((unsigned)e);
        }
        WAIT_V(0);
        if (early) st_sys32(fp, call);
        while (true) {
            const unsigned info = cur;
            char* const m = ml + par * SLOT;
            par ^= 1;
            const bool vB = jB < (unsigned)NMSG, vC = jC < (unsigned)NMSG;
            if (vB && eBok) fetch_msg(eBi, par);
            u64 eC = 0; unsigned t = 0;
            if (vC) {
                asm volatile("global_load_dwordx2 %0, %1, off sc0 sc1" : "=v"(eC) : "v"(qrow + jC * 8) : "memory");
                draw(t, ctr);
            }
            const int msg = info >> 22;
            int ti, tj; tile_of(msg >> 3, ti, tj);
            float* o = P.out + (size_t)(2048 * (src >> 2) + 128 * ti + 32 * ((msg & 7) >> 1) + 4 * (lane >> 4)) * GN
                       + 1024 * (src & 3) + 128 * tj + 64 * (msg & 1) + 4 * (lane & 15);
            const unsigned L = min(info & 4095u, 2048u) >> 5, ne = (info >> 12) & 127u;
            const unsigned mask = *(const unsigned*)(m + lane * 4);
            if (ne < 64u) {
                const unsigned esc = *(const unsigned short*)(m + 256 + 104 * L + 4 * (info & 31u) + 2 * lane);
                const unsigned rem = *(const unsigned*)(m + 256 + 104 * L + 4 * (lane & 31));
                const unsigned add = (unsigned)__builtin_amdgcn_readfirstlane(esc) << 23;
                asm volatile("" ::: "memory");
                *(unsigned*)(m + 256 + 128 * L + 4 * (lane & 31)) = (rem & 0x03ffffffu) + add;
                if (lane < L) {
                    char* pl = m + 256 + lane * 8;
                    u32x2 w[13], h[3];
#pragma unroll
                    for (int p = 0; p < 13; ++p) w[p] = *(const u32x2*)(pl + p * 8 * L);
#pragma unroll
                    for (int k = 0; k < 3; ++k) {
                        h[k] = (w[12] >> (2 + 2 * k)) & 0x03000000u;
#pragma unroll
                        for (int c = 0; c < 4; ++c) h[k] |= (w[4 * k + c] >> 26) << (6 * c);
                        *(u32x2*)(pl + (13 + k) * 8 * L) = h[k] + add;
                    }
#pragma unroll
                    for (int p = 0; p < 13; ++p) *(u32x2*)(pl + p * 8 * L) = (w[p] & 0x03ffffffu) + add;
                }
                if (lane - 1u < ne)
                    __hip_atomic_fetch_add((unsigned*)(m + 256 + 4 * (esc & 2047u)), (32u | (esc >> 11)) << 26, __ATOMIC_RELAXED, __HIP_MEMORY_SCOPE_WAVEFRONT);
                asm volatile("" ::: "memory");
            }
            unsigned pre, tot; wave_scan6(__builtin_popcount(mask), pre, tot);
            unsigned pos = 256 + pre * 4;
            u32x4 v[8];
#pragma unroll
            for (int e = 0; e < 32; ++e) {
                v[e >> 2][e & 3] = *(const unsigned*)(m + pos);
                pos += ((mask >> e) & 1u) * 4u;
            }
            asm volatile("" ::: "memory");
#pragma unroll
            for (int e4 = 0; e4 < 8; ++e4) {
#pragma unroll
                for (int c = 0; c < 4; ++c)
                    v[e4][c] &= (unsigned)(((int)(mask << (31 - (4 * e4 + c)))) >> 31);
                st_wt(o + (size_t)((e4 >> 2) * 16 + (e4 & 3)) * GN, v[e4]);
            }
            asm volatile("s_waitcnt vmcnt(8)" : "+v"(eC), "+v"(t) :: "memory");
            if (!vB) break;
            if (!eBok) { eBi = poll(jB); fetch_msg(eBi, par); WAIT_V(0); }
            cur = eBi;
            jB = jC;
            eBok = __builtin_amdgcn_readfirstlane((unsigned)(eC >> 32)) >= call; eBi = __builtin_amdgcn_readfirstlane((unsigned)eC);
            jC = vC ? __builtin_amdgcn_readfirstlane(t) - base : ~0u;
        }
        if (bad) { WAIT_V(0); P.out[(size_t)(2048 * (src >> 2) + 2047) * GN + 1024 * (src & 3) + lane] = NAN; }
    }
}

extern "C" void kernel_launch(int rank, int nranks, ncclComm_t comm,
                              hipStream_t stream,
                              void* const* d_in, const int* in_sizes, int n_in,
                              void* d_out, int out_size,
                              const int* in_dtypes, int out_dtype,
                              void* const* d_out_all,
                              void* const* d_ws_all, size_t ws_bytes) {
    if (nranks != NR || (unsigned)rank >= (unsigned)NR || ws_bytes < WS_NEED || n_in < 2 ||
        in_sizes[0] != 4096 * KL || in_sizes[1] != KL * GN || out_size != 4096 * GN) {
        fprintf(stderr, "unexpected problem shape\n");
        return;
    }
    HIP_CHECK(hipFuncSetAttribute((const void*)gemm_ar_fused, hipFuncAttributeMaxDynamicSharedMemorySize, LDS_BYTES));
    Params P;
    P.x = (const float*)d_in[0];
    P.w = (const float*)d_in[1];
    P.out = (float*)d_out;
    for (int p = 0; p < NR; ++p) P.ws[p] = (char*)d_ws_all[p];
    P.rank = rank;
    gemm_ar_fused<<<NWG, NTHR, LDS_BYTES, stream>>>(P);
    HIP_CHECK(hipGetLastError());
}
```

```cpp
#include <hip/hip_runtime.h>
#include <rccl/rccl.h>
#include <cstdio>
#include <cstdint>

typedef unsigned u32x4 __attribute__((ext_vector_type(4)));
typedef unsigned u32x2 __attribute__((ext_vector_type(2)));
typedef unsigned short u16x2 __attribute__((ext_vector_type(2)));
typedef float f32x4 __attribute__((ext_vector_type(4)));
typedef float f32x2 __attribute__((ext_vector_type(2)));
typedef _Float16 f16x8 __attribute__((ext_vector_type(8)));
typedef unsigned long long u64;

#define HIP_CHECK(expr) do { hipError_t _e = (expr); if (_e != hipSuccess) { \
    fprintf(stderr, "HIP error %s at %s:%d\n", hipGetErrorString(_e), __FILE__, __LINE__); \
    return; } } while (0)

constexpr int NR = 8;
constexpr int GN = 4096;
constexpr int KL = 512;
constexpr int NWG = 256, NTHR = 512;
#define NP_ 28
#define NPI_ 26
#define DLY0 64
#define DLYA 150
#define DLYS 2
#define DLYM 600
#define PSL 16
#define PHOLD 400
#define AFG 48
#define AFS 1
#define WPB 2
constexpr int NP = NP_;
constexpr int NPI = NPI_;
constexpr int NSEND = 112;
constexpr int NGEMM = 128;
constexpr int NMSG = NGEMM * 8;
constexpr int NPIECE = 768;
constexpr size_t UNIT = 16384;
constexpr size_t SLOT = 9216;
constexpr unsigned long long DEADLINE = 40000000ull;
#define PUSHB 1
#define NE 8
constexpr int NPUSH = (16 + NE) * PUSHB;
constexpr int P0 = 4 * (8 + NE) * PUSHB;


static_assert(NPUSH == 24 && WPB == 2 && NP <= 32 && NPI > 16 && NPI <= NP && AFG % 16 == 0 && AFG >= NPUSH, "layout");

constexpr size_t OFF_CNT   = 57344;
constexpr size_t OFF_TMO   = 4096;
constexpr size_t OFF_TICK  = 6144;
constexpr size_t OFF_LF    = 16384;
constexpr size_t OFF_AF    = 49152;
constexpr size_t OFF_AC    = 12288;
constexpr size_t OFF_JT    = 8192;
constexpr size_t OFF_Q     = 65536;
constexpr size_t OFF_X     = 1u << 20;
constexpr size_t OFF_S     = OFF_X + 24 * 64 * UNIT;
constexpr size_t OFF_O     = OFF_S + 512 * UNIT;
constexpr size_t OFF_R     = OFF_O + (size_t)NMSG * SLOT;
constexpr size_t WS_NEED   = OFF_R + 512 * UNIT;

constexpr int LDS_STAGE = 32768, LDS_MISC = 4 * LDS_STAGE, LDS_SNAP = LDS_MISC + 1024, LDS_BYTES = LDS_SNAP + 2048;
constexpr int WAVE_LDS = 16384;

struct Params { const float* x; const float* w; float* out; char* ws[NR]; int rank; };

__device__ __forceinline__ char* ws_sel(const Params& P, int d) {
    char* r = P.ws[0];
#pragma unroll
    for (int p = 1; p < NR; ++p) { char* w = P.ws[p]; asm("" : "+s"(w)); r = d == p ? w : r; }
    return r;
}
__device__ __forceinline__ void st_wt(void* p, u32x4 v) {
    asm volatile("global_store_dwordx4 %0, %1, off sc0 sc1\n\ts_nop 1" :: "v"(p), "v"(v) : "memory");
}
__device__ __forceinline__ void glds1(const void* g, unsigned lds) {
    unsigned keep;
    asm volatile("s_mov_b32 %0, m0\n\ts_mov_b32 m0, %2\n\ts_nop 0\n\t"
                 "global_load_lds_dwordx4 %1, off sc0 sc1\n\ts_mov_b32 m0, %0"
                 : "=&s"(keep) : "v"(g), "s"(lds) : "memory");
}
__device__ __forceinline__ void glds4(const void* g, unsigned lds) {
    unsigned keep;
    asm volatile("s_mov_b32 %0, m0\n\ts_mov_b32 m0, %2\n\ts_nop 0\n\t"
                 "global_load_lds_dwordx4 %1, off sc0 sc1\n\t"
                 "global_load_lds_dwordx4 %1, off offset:1024 sc0 sc1\n\t"
                 "global_load_lds_dwordx4 %1, off offset:2048 sc0 sc1\n\t"
                 "global_load_lds_dwordx4 %1, off offset:3072 sc0 sc1\n\t"
                 "s_mov_b32 m0, %0"
                 : "=&s"(keep) : "v"(g), "s"(lds) : "memory");
}
#define WAIT_V(n) asm volatile("s_waitcnt vmcnt(" #n ")" ::: "memory")
#define WAIT_L(n) asm volatile("s_waitcnt lgkmcnt(" #n ")" ::: "memory")
#define BAR() do { asm volatile("s_waitcnt lgkmcnt(0)" ::: "memory"); __builtin_amdgcn_s_barrier(); asm volatile("" ::: "memory"); } while (0)

__device__ __forceinline__ u64 ld_sys64(const void* p) {
    return __hip_atomic_load((const u64*)p, __ATOMIC_RELAXED, __HIP_MEMORY_SCOPE_SYSTEM);
}
__device__ __forceinline__ void st_sys64(void* p, u64 v) {
    asm volatile("global_store_dwordx2 %0, %1, off sc0 sc1\n\ts_nop 1" :: "v"(p), "v"(v) : "memory");
}
__device__ __forceinline__ unsigned ld_sys32(const void* p) {
    return __hip_atomic_load((const unsigned*)p, __ATOMIC_RELAXED, __HIP_MEMORY_SCOPE_SYSTEM);
}
__device__ __forceinline__ void st_sys32(void* p, unsigned v) {
    asm volatile("global_store_dword %0, %1, off sc0 sc1\n\ts_nop 1" :: "v"(p), "v"(v) : "memory");
}
__device__ __forceinline__ unsigned ld_ag32(const void* p) {
    return __hip_atomic_load((const unsigned*)p, __ATOMIC_RELAXED, __HIP_MEMORY_SCOPE_AGENT);
}
__device__ __forceinline__ void st_ag32(void* p, unsigned v) {
    __hip_atomic_store((unsigned*)p, v, __ATOMIC_RELAXED, __HIP_MEMORY_SCOPE_AGENT);
}
__device__ __forceinline__ bool expired(char* me, unsigned call, u64 t0) {
    const u64 dt = __builtin_amdgcn_s_memrealtime() - t0;
    if (dt < 100000) return false;
    bool e = ld_ag32(me + OFF_TMO) == call || dt > DEADLINE;
    e = __builtin_amdgcn_readfirstlane(e);
    if (e) st_ag32(me + OFF_TMO, call);
    return e;
}
__device__ __forceinline__ void wave_scan6(unsigned c, unsigned& pre, unsigned& tot) {
    pre = 0; tot = 0;
#pragma unroll
    for (int b = 0; b < 6; ++b) {
        u64 m = __builtin_amdgcn_ballot_w64((c >> b) & 1);
        unsigned p = __builtin_amdgcn_mbcnt_hi((unsigned)(m >> 32), __builtin_amdgcn_mbcnt_lo((unsigned)m, 0));
        pre += p << b;
        tot += (unsigned)__builtin_popcountll(m) << b;
    }
}
__device__ __forceinline__ void tile_of(int g, int& i, int& j) {
    int xcd = g & 7, t = g >> 3;
    i = 2 * (t >> 1) + (xcd >> 2);
    j = 2 * (xcd & 3) + (t & 1);
}

#define LAG 8
template <int NA, int NL> __device__ __forceinline__ void seq_of(int U, bool& late, int& ks, int& o) {
    if (U < NA * LAG) { ks = U / NA; o = U - NA * ks; late = false; }
    else if (U < NA * LAG + (NA + NL) * (8 - LAG)) {
        const int V = U - NA * LAG, s = V / (NA + NL); o = V - s * (NA + NL);
        late = o < NL;
        if (late) ks = s; else { ks = s + LAG; o -= NL; }
    } else {
        const int V = U - (NA * LAG + (NA + NL) * (8 - LAG)), s = V / NL;
        o = V - NL * s; ks = 8 - LAG + s; late = true;
    }
}
__device__ __forceinline__ void unit_of(int U, bool& isx, int& grp, int& strip, int& ks) {
    bool late; int o; seq_of<32 + 2 * NE, 32 - 2 * NE>(U, late, ks, o);
    isx = late || o < 2 * NE;
    if (isx) { grp = o & 1; strip = (late ? NE : 0) + (o >> 1); }
    else     { o -= 2 * NE; grp = ((o >> 3) + o) & 3; strip = o & 7; }
}
__device__ __forceinline__ unsigned msg_quads(unsigned info) {
    const unsigned n = min(info & 4095u, 2048u), ne = (info >> 12) & 127u;
    return 16 + (ne > 63u ? (n + 3) >> 2 : (104 * (n >> 5) + 4 * (n & 31u) + 2 * ne + 17) >> 4);
}
__device__ __forceinline__ int link_unit(int L, int rank, int& U) {
    bool late; int ks, o; seq_of<8 + NE, 16 - NE>(L, late, ks, o);
    const int na = late ? min(ks + LAG, 8) : ks, nl = late ? ks : max(ks + 1 - LAG, 0);
    U = na * (32 + 2 * NE) + nl * (32 - 2 * NE)
        + (late || o < NE ? 2 * o + (rank >> 2) : NE + o + 8 * ((rank - o + NE) & 3));
    return (late ? NE + o : o < NE ? o : 16 - NE + o) * 64 + ks;
}

__global__ __launch_bounds__(NTHR, 2) void gemm_ar_fused(Params P) {
    extern __shared__ __attribute__((aligned(16))) char smem[];
    const int tid = threadIdx.x, lane = tid & 63;
    const int wv = __builtin_amdgcn_readfirstlane(tid >> 6);
    const int bid = blockIdx.x, b = (bid - (128 - NSEND)) & 255, rank = P.rank;
    char* const me = P.ws[rank];
    const unsigned lds0 = (unsigned)(uintptr_t)(__attribute__((address_space(3))) char*)smem;
    const u64 t0 = __builtin_amdgcn_s_memrealtime();

    unsigned* misc = (unsigned*)(smem + LDS_MISC);
    unsigned cnt = ld_ag32(me + OFF_CNT + 4 * (bid * 8 + wv));
    if (tid < 4) misc[tid] = 0;
    if (tid < 128) *(u32x4*)(smem + LDS_SNAP + tid * 16) = (u32x4){0u, 0u, 0u, 0u};
    BAR();
    const bool pblk = b < NSEND;
    const int cb = pblk ? bid + 128 : bid < 128 ? bid : bid >= 248 ? bid - 232 : bid - 104;
    if (cb >= DLY0 && !(pblk && wv < WPB))
        while (__builtin_amdgcn_s_memrealtime() - t0 < (unsigned)min(DLYA + DLYS * (cb - DLY0), DLYM)) __builtin_amdgcn_s_sleep(4);
    for (int p = pblk ? wv - WPB : wv; p >= 0 && p < 8; p += pblk ? 8 - WPB : 8) {
        const int pc = cb < NPUSH ? p * NPUSH + cb : cb * 8 + p;
        const int U = pc >> 2, qd = pc & 3;
        bool isx; int grp, strip, ks; unit_of(U, isx, grp, strip, ks);
        f16x8 c[4];
        if (isx) {
            const int kg = (lane & 7) ^ (lane >> 3);
            const float* s = P.x + (size_t)(2048 * grp + 128 * strip + 32 * qd + (lane >> 3)) * KL + 64 * ks + 4 * kg;
            f32x4 f[8];
#pragma unroll
            for (int t = 0; t < 8; ++t) f[t] = *(const f32x4*)(s + (t >> 1) * 8 * KL + (t & 1) * 32);
#pragma unroll
            for (int e = 0; e < 32; ++e) c[e >> 3][e & 7] = (_Float16)f[e >> 2][e & 3];
        } else {
            const float* s = P.w + (size_t)(64 * ks + 8 * qd) * GN + 1024 * grp + 128 * strip + 4 * (lane & 31) + 2 * (lane >> 5);
            f32x2 f[16];
#pragma unroll
            for (int e = 0; e < 16; ++e) f[e] = *(const f32x2*)(s + (size_t)((e & 3) + ((e >> 2) & 1) * 32 + (e >> 3) * 4) * GN);
#pragma unroll
            for (int e = 0; e < 32; ++e) c[e >> 3][e & 7] = (_Float16)f[(e >> 4) * 8 + (e & 7)][(e >> 3) & 1];
        }
        asm volatile("" : "+v"(cnt));
        const unsigned call = cnt + 1;
        const int st = isx ? strip : 16 + strip;
        const size_t li = (size_t)(st * 64 + rank * 8 + ks) * 4 + qd;
        const bool push = cb < NPUSH;
        const bool mine = grp == (isx ? rank >> 2 : rank & 3);
        if (push) {
#pragma unroll
            for (int h = 0; h < 4; ++h) {
                for (int t = 0; t < (isx ? 4 : 2); ++t) {
                    const unsigned d = (unsigned)(isx ? 4 * grp + ((t + qd) & 3) : grp + 4 * ((t + qd) & 1)) & (NR - 1);
                    st_wt(P.ws[d] + OFF_X + li * 4096 + lane * 16 + h * 1024, __builtin_bit_cast(u32x4, c[h]));
                }
            }
        } else {
            const size_t so = (size_t)(((isx ? grp * 16 + strip : 32 + grp * 8 + strip) * 8 + ks) * 4 + qd) * 4096;
            unsigned d[16], T[2] = {0u, 0u};
            u16x2 mx = {0, 0};
#pragma unroll
            for (int j = 0; j < 16; ++j) {
                d[j] = __builtin_bit_cast(u32x4, c[j >> 2])[j & 3];
                mx = __builtin_elementwise_max(mx, __builtin_bit_cast(u16x2, d[j] & 0x7fff7fffu));
            }
            unsigned em = max(mx.x, mx.y);
#pragma unroll
            for (int o = 32; o > 0; o >>= 1) em = max(em, (unsigned)__shfl_xor((int)em, o));
            const unsigned low = max((unsigned)__builtin_amdgcn_readfirstlane(em) >> 10, 7u) - 7u;
#pragma unroll
            for (int j = 0; j < 16; ++j) {
                d[j] = __builtin_bit_cast(unsigned, __builtin_bit_cast(u16x2, d[j]) - __builtin_bit_cast(u16x2, low * 0x04000400u));
                T[j >> 3] |= ((d[j] >> 13) & 0x00030003u) << (2 * (j & 7));
            }
            u64 z = ((u64)T[1] << 32) | T[0];
            u64 nz = (z | (z >> 1)) & 0x5555555555555555ull;
            unsigned epre, nesc; wave_scan6(__builtin_popcountll(nz), epre, nesc);
            char* el = smem + LDS_MISC - 2048 + wv * 256;
            if (nesc > 63u) {
#pragma unroll
                for (int h = 0; h < 4; ++h) st_wt(me + OFF_R + so + h * 1024 + lane * 16, __builtin_bit_cast(u32x4, c[h]));
            } else {
                for (; nz; nz &= nz - 1) {
                    const int bit = __builtin_ctzll(nz);
                    *(unsigned short*)(el + 2 * ++epre) = (lane << 7) | (bit << 1) | ((unsigned)(z >> bit) & 3u);
                }
#pragma unroll
                for (int j = 0; j < 14; ++j) {
                    const unsigned g = d[14 + j / 7], p = j % 7;
                    const unsigned top = p < 6 ? g << (13 - 2 * p) : ((g << 1) & 0x20002000u) | ((g >> 1) & 0x40004000u);
                    d[j] = (d[j] & 0x9fff9fffu) | (top & 0x60006000u);
                }
#pragma unroll
                for (int h = 0; h < 3; ++h)
                    st_wt(me + OFF_S + so + h * 1024 + lane * 16, (u32x4){d[4 * h], d[4 * h + 1], d[4 * h + 2], d[4 * h + 3]});
                st_sys64(me + OFF_S + so + 3072 + lane * 8, ((u64)d[13] << 32) | d[12]);
            }
            if (lane == 0) *(unsigned short*)el = min(nesc, 64u) | (low << 8);
            asm volatile("" ::: "memory");
            if (lane < 8) st_wt(me + OFF_S + so + 3584 + lane * 16, *(const u32x4*)(el + lane * 16));
            if (mine) {
#pragma unroll
                for (int h = 0; h < 4; ++h) st_wt(me + OFF_X + li * 4096 + lane * 16 + h * 1024, __builtin_bit_cast(u32x4, c[h]));
            }
        }
        WAIT_V(0);
        if (push) {
            if (lane < (isx ? 4 : 2)) st_sys32(ws_sel(P, isx ? 4 * grp + lane : grp + 4 * lane) + OFF_LF + li * 4, call);
        } else {
            unsigned k = 0;
            if (lane == 0) k = __hip_atomic_fetch_add(&misc[1], 1u, __ATOMIC_RELAXED, __HIP_MEMORY_SCOPE_WORKGROUP);
            if (__builtin_amdgcn_readfirstlane(k) == 7) {
                if (cb < AFG) {
                    if (lane < NR && lane != rank) st_sys32(ws_sel(P, lane) + OFF_AF + (rank * 256 + cb) * 4, call);
                } else {
                    if (lane == 0) k = __hip_atomic_fetch_add((unsigned*)(me + OFF_AC + (cb >> 4) * 128), 1u, __ATOMIC_RELAXED, __HIP_MEMORY_SCOPE_SYSTEM);
                    if ((__builtin_amdgcn_readfirstlane(k) & 15) == 15 && lane < 4 * NR && (lane >> 2) != rank)
                        st_wt(ws_sel(P, lane >> 2) + OFF_AF + (rank * 256 + (cb & ~15)) * 4 + (lane & 3) * 16, (u32x4){call, call, call, call});
                }
            }
            if (mine && lane == 0) st_sys32(me + OFF_LF + li * 4, call);
        }
    }
    const unsigned call = __builtin_amdgcn_readfirstlane(cnt) + 1;
    if (lane == 0) st_sys32(me + OFF_CNT + 4 * (bid * 8 + wv), call);

    if (bid >= 128) {
        const int g = bid & 127;
        int ti, tj; tile_of(g, ti, tj);
        const size_t fo = (size_t)((rank + lane) & 7) * 128 + (lane >> 3) * 16;
        const char* fw = me + OFF_LF + (size_t)(16 + tj) * 1024;
        const char* fx = me + OFF_LF + (size_t)ti * 1024;
        u64 avail = 0, left = ~0ull;
        constexpr int nst = 64;
        auto arrived = [&](u32x4 a, u32x4 c) {
            return left & __builtin_amdgcn_ballot_w64(min(min(min(a.x, a.y), min(a.z, a.w)), min(min(c.x, c.y), min(c.z, c.w))) >= call);
        };
        auto pick = [&](int kk) {
            if (!avail) avail = arrived(*(const u32x4*)(smem + LDS_SNAP + fo), *(const u32x4*)(smem + LDS_SNAP + 1024 + fo));
            while (!avail) {
                u32x4 a, c;
                asm volatile("global_load_dwordx4 %0, %2, off sc0 sc1\n\tglobal_load_dwordx4 %1, %3, off sc0 sc1\n\ts_waitcnt vmcnt(0)"
                             : "=&v"(a), "=&v"(c) : "v"(fw + fo), "v"(fx + fo) : "memory");
                avail = arrived(a, c);
                if (avail) break;
                if (expired(me, call, t0)) { avail = left; misc[2] = 1; break; }
                __builtin_amdgcn_s_sleep(PSL);
            }
            const int id = __builtin_ctzll(avail);
            avail &= avail - 1; left &= ~(1ull << id);
            if (lane == 0) misc[8 + (kk & 7)] = ((rank + id) & 7) * 8 + (id >> 3);
        };
        const int kh = wv >> 2, wr = (wv & 2) + kh, wc = wv & 1, q = lane >> 4, r = lane & 15;
        f32x4 acc[2][4];
        {
            f32x4 ac[4][4];
#pragma unroll
            for (int mi = 0; mi < 4; ++mi)
#pragma unroll
                for (int nn = 0; nn < 4; ++nn) ac[mi][nn] = (f32x4){0.f, 0.f, 0.f, 0.f};
            const char* gsrc = me + OFF_X + (size_t)(wv < 4 ? ti : 16 + tj) * 64 * UNIT
                               + (wv & 3) * 4096 + lane * 16;
            const unsigned ldst = lds0 + wv * 4096;
            auto stage = [&](int kk) {
                if (wv == 0) glds1(((kk & 1) ? fw : fx) + lane * 16, lds0 + LDS_SNAP + (kk & 1) * 1024);
                glds4(gsrc + (size_t)(misc[8 + (kk & 7)] & 63u) * UNIT, ldst + (kk & 3) * LDS_STAGE);
            };
            for (int kk = 0; kk < 3; ++kk) {
                if (wv == 0) pick(kk);
                BAR();
                stage(kk);
            }
            const int aoff = (32 * (wv & 2) + r) * 128 + ((4 * kh + q) ^ (r & 7)) * 16;
            const int boff = 16384 + kh * 8192 + q * 2048 + (16 * wc + r) * 16;
            for (int kk = 0; kk < nst; ++kk) {
                if (wv == 0 && kk + 3 < nst) pick(kk + 3);
                if (kk + 2 < nst) { if (wv == 0) WAIT_V(10); else WAIT_V(8); }
                else if (kk + 1 < nst) WAIT_V(4); else WAIT_V(0);
                BAR();
                if (kk + 3 < nst) stage(kk + 3);
                const char* st = smem + (kk & 3) * LDS_STAGE;
                f16x8 af[4], bf[4];
#pragma unroll
                for (int i = 0; i < 4; ++i) {
                    af[i] = *(const f16x8*)(st + aoff + i * 2048);
                    bf[i] = *(const f16x8*)(st + boff + (i & 1) * 1024 + (i >> 1) * 512);
                }
#pragma unroll
                for (int mi = 0; mi < 4; ++mi)
#pragma unroll
                    for (int nn = 0; nn < 4; ++nn)
                        ac[mi][nn] = __builtin_amdgcn_mfma_f32_16x16x32_f16(af[mi], bf[nn], ac[mi][nn], 0, 0, 0);
            }
            BAR();
#pragma unroll
            for (int i = 0; i < 8; ++i)
                *(f32x4*)(smem + (wv ^ 4) * WAVE_LDS + i * 1024 + lane * 16) = kh ? ac[i >> 2][i & 3] : ac[2 + (i >> 2)][i & 3];
            BAR();
#pragma unroll
            for (int i = 0; i < 8; ++i)
                acc[i >> 2][i & 3] = (kh ? ac[2 + (i >> 2)][i & 3] : ac[i >> 2][i & 3])
                                     + *(const f32x4*)(smem + wv * WAVE_LDS + i * 1024 + lane * 16);
        }
        const bool poison = misc[2];
        unsigned qi = 0;
        if (tid == 0) asm volatile("global_atomic_add %0, %1, %2, off sc0 sc1" : "=v"(qi) : "v"(me + OFF_TICK), "v"(1u) : "memory");
        if (poison) {
#pragma unroll
            for (int i = 0; i < 8; ++i) acc[i >> 2][i & 3] = (f32x4){NAN, NAN, NAN, NAN};
        }
        unsigned u[32], mask = 0, umax = 0;
#pragma unroll
        for (int mi = 0; mi < 2; ++mi)
#pragma unroll
            for (int j = 0; j < 4; ++j)
#pragma unroll
                for (int nn = 0; nn < 4; ++nn) {
                    const int e = (mi * 4 + j) * 4 + nn;
                    const float v = acc[mi][nn][j];
                    const bool keep = !(v <= 0.f);
                    u[e] = keep ? __float_as_uint(v) : 0u;
                    mask |= (keep ? 1u : 0u) << e;
                    umax = max(umax, u[e]);
                }
#pragma unroll
        for (int o = 32; o > 0; o >>= 1) umax = max(umax, (unsigned)__shfl_xor((int)umax, o));
        umax = __builtin_amdgcn_readfirstlane(umax);
        const unsigned ebase = max(umax >> 23, 7u) - 7u;
        unsigned em = 0;
#pragma unroll
        for (int e = 0; e < 32; ++e) em |= (u[e] < (ebase << 23) ? 1u : 0u) << e;
        em &= mask;
        unsigned pre, n, epre, nesc;
        wave_scan6(__builtin_popcount(mask), pre, n);
        wave_scan6(__builtin_popcount(em), epre, nesc);
        const bool raw = nesc > 63u || (umax >> 31);
        const unsigned sub = raw ? 0u : ebase << 23;
        char* ml = smem + wv * WAVE_LDS;
        *(unsigned*)(ml + lane * 4) = mask;
        {
            unsigned pos = 256 + pre * 4;
#pragma unroll
            for (int e = 0; e < 32; ++e) {
                const bool k = (mask >> e) & 1;
                *(unsigned*)(ml + (k ? pos : 12288u + lane * 4)) = u[e] - sub;
                pos += k ? 4 : 0;
            }
        }
        if (!raw) {
            const unsigned L = n >> 5;
            if (lane == 0) *(unsigned short*)(ml + 13312) = ebase;
            for (; em; em &= em - 1) {
                const unsigned pos = pre + __builtin_popcount(mask & ((em & -em) - 1u));
                *(unsigned short*)(ml + 13312 + 2 * ++epre) = pos | ((*(const unsigned*)(ml + 256 + 4 * pos) >> 26) << 11);
            }
            if (lane < L) {
                char* pl = ml + 256 + lane * 8;
                u32x2 h[3];
#pragma unroll
                for (int k = 0; k < 3; ++k) h[k] = *(const u32x2*)(pl + (13 + k) * 8 * L);
#pragma unroll
                for (int p = 0; p < 13; ++p) {
                    u32x2* a = (u32x2*)(pl + p * 8 * L);
                    const u32x2 top = p < 12 ? h[p >> 2] << (26 - 6 * (p & 3))
                                             : ((h[0] << 2) & 0x0c000000u) | ((h[1] << 4) & 0x30000000u) | ((h[2] << 6) & 0xc0000000u);
                    *a = (*a & 0x03ffffffu) | (top & 0xfc000000u);
                }
            }
            asm volatile("" ::: "memory");
            *(unsigned*)(ml + 256 + 104 * L + 4 * (lane & 31)) = *(const unsigned*)(ml + 256 + 128 * L + 4 * (lane & 31));
            asm volatile("" ::: "memory");
            *(unsigned short*)(ml + 256 + 104 * L + 4 * (n & 31u) + 2 * lane) = *(const unsigned short*)(ml + 13312 + 2 * lane);
        }
        const unsigned info = n | ((raw ? 64u : nesc) << 12) | ((unsigned)(g * 8 + wr * 2 + wc) << 22);
        const unsigned Q = msg_quads(info);
        const int msg = g * 8 + wr * 2 + wc;
        {
            char* dst = me + OFF_O + (size_t)msg * SLOT + lane * 16;
            for (unsigned i = 0; i < Q; i += 64)
                if (i + lane < Q) st_wt(dst + i * 16, *(const u32x4*)(ml + (i + lane) * 16));
        }
        asm volatile("s_waitcnt vmcnt(0)" : "+v"(qi) :: "memory");
        unsigned k = 0;
        if (tid == 0) misc[24] = qi;
        if (lane == 0) {
            misc[16 + wv] = info;
            asm volatile("" ::: "memory");
            k = __hip_atomic_fetch_add(&misc[3], 1u, __ATOMIC_RELAXED, __HIP_MEMORY_SCOPE_WORKGROUP);
        }
        if (__builtin_amdgcn_readfirstlane(k) == 7 && (lane >> 3) != rank) {
            const volatile unsigned* mv = misc;
            st_sys64(ws_sel(P, lane >> 3) + OFF_Q + (size_t)(rank * NMSG + (mv[24] & (NGEMM - 1)) * 8 + (lane & 7)) * 8,
                     ((u64)call << 32) | mv[16 + (lane & 7)]);
        }
        {
            float* o = P.out + (size_t)(2048 * (rank >> 2) + 128 * ti + 32 * wr + 4 * q) * GN
                       + 1024 * (rank & 3) + 128 * tj + 64 * wc + 4 * r;
#pragma unroll
            for (int e4 = 0; e4 < 8; ++e4)
                st_wt(o + (size_t)((e4 >> 2) * 16 + (e4 & 3)) * GN, (u32x4){u[4 * e4], u[4 * e4 + 1], u[4 * e4 + 2], u[4 * e4 + 3]});
        }
        return;
    }
    const int pj = b / 7 + 16 * wv, pw = 7 * pj + (b + wv) % 7;
    if (!pblk) return;
    if (wv >= WPB || pj >= NP) return;

    const unsigned src = (unsigned)(rank + 1 + pw - 7 * pj) & (NR - 1);
    char* const peer = P.ws[src];
    char* const ml = smem + wv * 2 * SLOT;
    const unsigned mlds = lds0 + wv * 2 * SLOT;
    bool bad = false;
    auto draw = [&](unsigned& t, const char* ctr) {
        if (lane == 0) asm volatile("global_atomic_add %0, %1, %2, off sc0 sc1" : "=v"(t) : "v"(ctr), "v"(1u) : "memory");
    };
    const char* const qrow = me + OFF_Q + (size_t)src * NMSG * 8;
    const unsigned ibase = (call - 1) * (unsigned)(NPIECE - P0 - NPI) - (P0 + 2 * NPI);
    auto fetch_msg = [&](unsigned info, unsigned p) {
        const unsigned Q = msg_quads(info);
        const char* s = peer + OFF_O + (size_t)(info >> 22) * SLOT + lane * 16;
        for (unsigned i = 0; i < Q; i += 64)
            if (i + lane < Q) glds1(s + i * 16, mlds + p * SLOT + i * 16);
    };
    unsigned cur = 0, par = 0;
    bool early = false;
    char* fp = me + 256;
    if (pj >= NPI) {
        while (ld_sys32(me + OFF_JT + src * 128) - ibase < (unsigned)NPIECE) {
            if (expired(me, call, t0)) { bad = true; break; }
            __builtin_amdgcn_s_sleep(32);
        }
    } else {
        const char* ctr = me + OFF_JT + src * 128;
        u64 ok = 0;
        while (__builtin_amdgcn_s_memrealtime() - t0 < PHOLD) __builtin_amdgcn_s_sleep(2);
        auto fetch = [&](int pi, unsigned p, bool all) {
            int U;
            const int lu = link_unit(pi >> 2, rank, U), st = lu >> 6;
            for (; !((ok >> (U >> 3)) & 1); all = false) {
                u32x4 a = {0u, 0u, 0u, 0u};
                if (all || lane == (U >> 3))
                    asm volatile("global_load_dwordx4 %0, %1, off sc0 sc1\n\ts_waitcnt vmcnt(0)"
                                 : "+&v"(a) : "v"(me + OFF_AF + src * 1024 + lane * 16) : "memory");
                ok |= __builtin_amdgcn_ballot_w64(min(min(a.x, a.y), min(a.z, a.w)) >= call);
                const unsigned own = U & 4 ? (U & 2 ? a.w : a.z) : (U & 2 ? a.y : a.x);
                if ((unsigned)__builtin_amdgcn_readlane(own, U >> 3) >= call) break;
                if (expired(me, call, t0)) { bad = true; ok = ~0ull; }
                __builtin_amdgcn_s_sleep(AFS);
            }
            const unsigned sidx = (unsigned)((st < 16 ? (rank >> 2) * 16 + st : 16 + (rank & 3) * 8 + st) * 8 + (lu & 7)) * 4 + (pi & 3);
            const char* s = peer + OFF_S + (size_t)sidx * 4096 + lane * 16;
            glds1(s, mlds + p * SLOT); glds1(s + 1024, mlds + p * SLOT + 1024); glds1(s + 2048, mlds + p * SLOT + 2048);
            if (lane < 40) glds1(s + 3072, mlds + p * SLOT + 3072);
            return (((unsigned)(lu + src * 8) * 4 + (pi & 3)) << 16) | sidx;
        };
        unsigned nxt = P0 + NPI + pj;
        auto peek = [&] { if (lane == 0) glds1(qrow + (pj >> 1) * 16, lds0 + 65536 + wv * 16); };
        auto first = [&] {
            const u32x2 e = *(const volatile u32x2*)(smem + 65536 + wv * 16 + (pj & 1) * 8);
            early = (unsigned)__builtin_amdgcn_readfirstlane(e.y) >= call;
            if (early) { cur = __builtin_amdgcn_readfirstlane(e.x); fetch_msg(cur, par); }
        };
        cur = fetch(P0 + pj, 0, false);
        {
            u32x4 a;
            asm volatile("global_load_dwordx4 %0, %1, off sc0 sc1\n\ts_waitcnt vmcnt(0)"
                         : "=&v"(a) : "v"(me + OFF_AF + src * 1024 + lane * 16) : "memory");
            ok = __builtin_amdgcn_ballot_w64(min(min(a.x, a.y), min(a.z, a.w)) >= call);
        }
        while (true) {
            char* const m = ml + par * SLOT;
            const unsigned id = cur;
            const bool more = nxt < (unsigned)NPIECE;
            unsigned t = 0;
            par ^= 1;
            if (more) { cur = fetch(nxt, par, true); draw(t, ctr); }
            else first();
            if (!early) peek();
            const unsigned esc = *(const unsigned short*)(m + 3584 + 2 * lane);
            *(u64*)(m + 4096 + lane * 8) = 0;
            u32x4 v[4];
#pragma unroll
            for (int h = 0; h < 3; ++h) v[h] = *(const u32x4*)(m + h * 1024 + lane * 16);
            v[3].xy = *(const u32x2*)(m + 3072 + lane * 8);
            const unsigned hdr = __builtin_amdgcn_readfirstlane(esc);
            if (hdr & 64u) {
                WAIT_L(0);
                glds4(peer + OFF_R + (size_t)(id & 2047u) * 4096 + lane * 16, mlds + (par ^ 1) * SLOT);
                WAIT_V(0);
#pragma unroll
                for (int h = 0; h < 4; ++h) v[h] = *(const u32x4*)(m + h * 1024 + lane * 16);
            } else {
                if (lane - 1u < (hdr & 63u))
                    __hip_atomic_fetch_or((unsigned*)(m + 4096 + ((esc >> 6) & 127u) * 4), (esc & 3u) << ((esc >> 1) & 30u), __ATOMIC_RELAXED, __HIP_MEMORY_SCOPE_WAVEFRONT);
                asm volatile("" ::: "memory");
                const u32x2 T = *(const u32x2*)(m + 4096 + lane * 8);
#pragma unroll
                for (int k = 0; k < 2; ++k) {
                    const unsigned g = v[(7 * k + 6) >> 2][(7 * k + 6) & 3];
                    unsigned r = ((g >> 1) & 0x10001000u) | ((g << 1) & 0x80008000u);
#pragma unroll
                    for (int p = 0; p < 6; ++p) r |= (v[(7 * k + p) >> 2][(7 * k + p) & 3] >> (13 - 2 * p)) & (0x00030003u << (2 * p));
                    v[3][2 + k] = r;
                }
                const u16x2 add = __builtin_bit_cast(u16x2, (hdr >> 8) * 0x04000400u);
#pragma unroll
                for (int j = 0; j < 16; ++j) {
                    const unsigned up = (j & 7) < 7 ? T[j >> 3] << (13 - 2 * (j & 7)) : T[j >> 3] >> 1;
                    v[j >> 2][j & 3] = __builtin_bit_cast(unsigned,
                        __builtin_bit_cast(u16x2, (v[j >> 2][j & 3] & 0x9fff9fffu) | (up & 0x60006000u)) + add);
                }
            }
            if (more) WAIT_V(6);
            else if (early && msg_quads(cur) - 193u < 64u) WAIT_V(4);
            else {
                WAIT_V(0);
                if (!early) first();
            }
            st_sys32(fp, call);
            char* d = me + OFF_X + (size_t)(id >> 16) * 4096 + lane * 16;
#pragma unroll
            for (int h = 0; h < 4; ++h) st_wt(d + h * 1024, v[h]);
            fp = me + OFF_LF + (id >> 16) * 4;
            if (!more) break;
            asm volatile("s_waitcnt vmcnt(5)" : "+v"(t) :: "memory");
            nxt = __builtin_amdgcn_readfirstlane(t) - ibase;
        }
    }
    {
        const char* ctr = me + OFF_JT + 1024 + src * 128;
        const unsigned base = (call - 1) * (unsigned)(NMSG - 2 * NP) - 3 * NP;
        auto poll = [&](unsigned j) {
            while (true) {
                const u64 e = ld_sys64(qrow + j * 8);
                if (__builtin_amdgcn_readfirstlane((unsigned)(e >> 32)) >= call) return (unsigned)__builtin_amdgcn_readfirstlane((unsigned)e);
                if (bad || expired(me, call, t0)) { bad = true; return 0u; }
                __builtin_amdgcn_s_sleep(4);
            }
        };
        unsigned jB = NP + pj, jC = 2 * NP + pj, eBi = 0;
        bool eBok = false;
        if (!early) {
            WAIT_V(0);
            st_sys32(fp, call);
            cur = poll(pj);
            fetch_msg(cur, par);
        }
        {
            const u64 e = ld_sys64(qrow + jB * 8);
            eBok = __builtin_amdgcn_readfirstlane((unsigned)(e >> 32)) >= call; eBi = __builtin_amdgcn_readfirstlane((unsigned)e);
        }
        WAIT_V(0);
        if (early) st_sys32(fp, call);
        while (true) {
            const unsigned info = cur;
            char* const m = ml + par * SLOT;
            par ^= 1;
            const bool vB = jB < (unsigned)NMSG, vC = jC < (unsigned)NMSG;
            if (vB && eBok) fetch_msg(eBi, par);
            u64 eC = 0; unsigned t = 0;
            if (vC) {
                asm volatile("global_load_dwordx2 %0, %1, off sc0 sc1" : "=v"(eC) : "v"(qrow + jC * 8) : "memory");
                draw(t, ctr);
            }
            const int msg = info >> 22;
            int ti, tj; tile_of(msg >> 3, ti, tj);
            float* o = P.out + (size_t)(2048 * (src >> 2) + 128 * ti + 32 * ((msg & 7) >> 1) + 4 * (lane >> 4)) * GN
                       + 1024 * (src & 3) + 128 * tj + 64 * (msg & 1) + 4 * (lane & 15);
            const unsigned L = min(info & 4095u, 2048u) >> 5, ne = (info >> 12) & 127u;
            const unsigned mask = *(const unsigned*)(m + lane * 4);
            if (ne < 64u) {
                const unsigned esc = *(const unsigned short*)(m + 256 + 104 * L + 4 * (info & 31u) + 2 * lane);
                const unsigned rem = *(const unsigned*)(m + 256 + 104 * L + 4 * (lane & 31));
                const unsigned add = (unsigned)__builtin_amdgcn_readfirstlane(esc) << 23;
                asm volatile("" ::: "memory");
                *(unsigned*)(m + 256 + 128 * L + 4 * (lane & 31)) = (rem & 0x03ffffffu) + add;
                if (lane < L) {
                    char* pl = m + 256 + lane * 8;
                    u32x2 w[13], h[3];
#pragma unroll
                    for (int p = 0; p < 13; ++p) w[p] = *(const u32x2*)(pl + p * 8 * L);
#pragma unroll
                    for (int k = 0; k < 3; ++k) {
                        h[k] = (w[12] >> (2 + 2 * k)) & 0x03000000u;
#pragma unroll
                        for (int c = 0; c < 4; ++c) h[k] |= (w[4 * k + c] >> 26) << (6 * c);
                        *(u32x2*)(pl + (13 + k) * 8 * L) = h[k] + add;
                    }
#pragma unroll
                    for (int p = 0; p < 13; ++p) *(u32x2*)(pl + p * 8 * L) = (w[p] & 0x03ffffffu) + add;
                }
                if (lane - 1u < ne)
                    __hip_atomic_fetch_add((unsigned*)(m + 256 + 4 * (esc & 2047u)), (32u | (esc >> 11)) << 26, __ATOMIC_RELAXED, __HIP_MEMORY_SCOPE_WAVEFRONT);
                asm volatile("" ::: "memory");
            }
            unsigned pre, tot; wave_scan6(__builtin_popcount(mask), pre, tot);
            unsigned pos = 256 + pre * 4;
            u32x4 v[8];
#pragma unroll
            for (int e = 0; e < 32; ++e) {
                v[e >> 2][e & 3] = *(const unsigned*)(m + pos);
                pos += ((mask >> e) & 1u) * 4u;
            }
            asm volatile("" ::: "memory");
#pragma unroll
            for (int e4 = 0; e4 < 8; ++e4) {
#pragma unroll
                for (int c = 0; c < 4; ++c)
                    v[e4][c] &= (unsigned)(((int)(mask << (31 - (4 * e4 + c)))) >> 31);
                st_wt(o + (size_t)((e4 >> 2) * 16 + (e4 & 3)) * GN, v[e4]);
            }
            asm volatile("s_waitcnt vmcnt(8)" : "+v"(eC), "+v"(t) :: "memory");
            if (!vB) break;
            if (!eBok) { eBi = poll(jB); fetch_msg(eBi, par); WAIT_V(0); }
            cur = eBi;
            jB = jC;
            eBok = __builtin_amdgcn_readfirstlane((unsigned)(eC >> 32)) >= call; eBi = __builtin_amdgcn_readfirstlane((unsigned)eC);
            jC = vC ? __builtin_amdgcn_readfirstlane(t) - base : ~0u;
        }
        if (bad) { WAIT_V(0); P.out[(size_t)(2048 * (src >> 2) + 2047) * GN + 1024 * (src & 3) + lane] = NAN; }
    }
}

extern "C" void kernel_launch(int rank, int nranks, ncclComm_t comm,
                              hipStream_t stream,
                              void* const* d_in, const int* in_sizes, int n_in,
                              void* d_out, int out_size,
                              const int* in_dtypes, int out_dtype,
                              void* const* d_out_all,
                              void* const* d_ws_all, size_t ws_bytes) {
    if (nranks != NR || (unsigned)rank >= (unsigned)NR || ws_bytes < WS_NEED || n_in < 2 ||
        in_sizes[0] != 4096 * KL || in_sizes[1] != KL * GN || out_size != 4096 * GN) {
        fprintf(stderr, "unexpected problem shape\n");
        return;
    }
    HIP_CHECK(hipFuncSetAttribute((const void*)gemm_ar_fused, hipFuncAttributeMaxDynamicSharedMemorySize, LDS_BYTES));
    Params P;
    P.x = (const float*)d_in[0];
    P.w = (const float*)d_in[1];
    P.out = (float*)d_out;
    for (int p = 0; p < NR; ++p) P.ws[p] = (char*)d_ws_all[p];
    P.rank = rank;
    gemm_ar_fused<<<NWG, NTHR, LDS_BYTES, stream>>>(P);
    HIP_CHECK(hipGetLastError());
}
```
